```python
import numpy as np

kernel_name = "gpu_ag_gemm_m4096_k4096_n32768_f32_silu"


ABSMAX_RTOL = 0.001

COMPUTE_DTYPE = "float32"
EPILOGUE = "silu"
IN_QUANT = None
OUT_QUANT = None
IN_DTYPES = {"x": "float32", "w_mat": "float32"}
WORLD_SIZE = 8

def setup_inputs():

    x = np.random.randn(4096, 4096).astype(np.float32)
    w_mat = (np.random.randn(4096, 32768) / np.sqrt(4096)).astype(np.float32)
    return {"x": x, "w_mat": w_mat}

def _epilogue(y):
    return (y / (1.0 + np.exp(-np.clip(y, -60.0, 60.0)))).astype(np.float32)

def reference(x, w_mat):
    y = (x @ w_mat).astype(np.float32)
    return _epilogue(y)
```

```cpp
#include <hip/hip_runtime.h>
#include <rccl/rccl.h>
#include <cstdio>
#include <cstdint>

#define HIP_CHECK(expr) do { hipError_t _e = (expr); if (_e != hipSuccess) { \
    fprintf(stderr, "HIP error %s at %s:%d\n", hipGetErrorString(_e), __FILE__, __LINE__); \
    return; } } while (0)

#define LAS __attribute__((address_space(3)))
typedef _Float16 f16x8 __attribute__((ext_vector_type(8)));
typedef float f32x4 __attribute__((ext_vector_type(4)));
typedef unsigned u32x4 __attribute__((ext_vector_type(4)));

constexpr int NR = 8;
constexpr int KD = 4096;
constexpr int NLOC = 4096;
constexpr int MLOC = 512;
constexpr int BK = 64, HALF = 128;
constexpr int HTB = HALF * BK * 2;
constexpr int STAGE_BYTES = 8 * HTB;
constexpr int NT = KD / BK;
constexpr int NWG = 256;
constexpr int PF_DIST = 4;

constexpr size_t OFF_CTR_CALL = 0, OFF_CTR_X = 256, OFF_TMO = 512, OFF_FLAG = 4096;
constexpr size_t OFF_X16 = 1u << 20, X16_BYTES = 4u << 20;
constexpr size_t OFF_EX = 9u << 20, EX_BYTES = 4096;
constexpr size_t OFF_W16 = 16u << 20;
constexpr size_t OFF_EW = 48u << 20;
constexpr size_t WS_NEEDED = 49u << 20;

constexpr int LDS_EXP = STAGE_BYTES;
constexpr int LDS_PF = STAGE_BYTES + 2048;
constexpr int LDS_CTL = STAGE_BYTES + 4096;
constexpr int LDS_TOTAL = STAGE_BYTES + 4096 + 256;

constexpr unsigned long long DEADLINE_TICKS = 100000000ull;

struct Params {
    const float* x;
    const float* w;
    float* out;
    char* ws[NR];
    int rank;
};

__device__ __forceinline__ int lds_byte(int r, int c) {
    const int st = (r >> 4) * 2 + (c >> 5), rr = r & 15, cc = c & 31, ob = rr * 64 + cc * 2;
    return st * 1024 + (ob ^ (((ob >> 9) & 1) << 5));
}

__device__ __forceinline__ int scale_exp(float amax) {
    return amax > 0.f ? __builtin_amdgcn_frexp_expf(amax) - 15 : 0;
}

__device__ __forceinline__ unsigned pack2(float a, float b, int e) {
    const _Float16 ha = (_Float16)__builtin_amdgcn_ldexpf(a, -e), hb = (_Float16)__builtin_amdgcn_ldexpf(b, -e);
    return (unsigned)__builtin_bit_cast(unsigned short, ha) | ((unsigned)__builtin_bit_cast(unsigned short, hb) << 16);
}

__device__ __forceinline__ float silu(float y) {
    const float c = __builtin_fminf(__builtin_fmaxf(y, -60.f), 60.f);
    const float e = __builtin_amdgcn_exp2f(c * -1.44269504088896341f);
    return y * __builtin_amdgcn_rcpf(1.0f + e);
}

__device__ __forceinline__ void prep_phase(const Params& p, LAS char* lds) {
    const int tid = threadIdx.x, wid = tid >> 6, lane = tid & 63, b = blockIdx.x;
    char* ws = p.ws[p.rank];
    LAS float* red = (LAS float*)lds;
    LAS unsigned* ctl = (LAS unsigned*)(lds + 1024);
    if (tid == 0) {
        const unsigned t = __hip_atomic_fetch_add((unsigned*)(ws + OFF_CTR_CALL), 1u, __ATOMIC_RELAXED, __HIP_MEMORY_SCOPE_AGENT);
        ctl[0] = t / NWG + 1;
    }
    {
        const int row = 2 * b + (tid >> 8), kk = (tid & 255) * 16;
        const f32x4* src = (const f32x4*)(p.x + (size_t)row * KD + kk);
        f32x4 v[4];
#pragma unroll
        for (int i = 0; i < 4; ++i) v[i] = src[i];
        float m = 0.f;
#pragma unroll
        for (int i = 0; i < 4; ++i)
#pragma unroll
            for (int j = 0; j < 4; ++j) m = __builtin_fmaxf(m, __builtin_fabsf(v[i][j]));
#pragma unroll
        for (int s = 1; s < 64; s <<= 1) m = __builtin_fmaxf(m, __shfl_xor(m, s));
        if (lane == 0) red[wid] = m;
        __syncthreads();
        const int w0 = (tid >> 8) * 4;
        m = __builtin_fmaxf(__builtin_fmaxf(red[w0], red[w0 + 1]), __builtin_fmaxf(red[w0 + 2], red[w0 + 3]));
        const unsigned call = ctl[0];
        const int e = scale_exp(m);
        const __amdgpu_buffer_rsrc_t rs = __builtin_amdgcn_make_buffer_rsrc(ws, 0, (int)WS_NEEDED, 0x00020000);
        const unsigned ioff = (unsigned)(OFF_X16 + (call & 1) * X16_BYTES) + (unsigned)(kk >> 6) * 65536u + (unsigned)(row >> 7) * HTB;
        const int kl = kk & 63;
#pragma unroll
        for (int c = 0; c < 2; ++c) {
            u32x4 o;
            o.x = pack2(v[2 * c][0], v[2 * c][1], e); o.y = pack2(v[2 * c][2], v[2 * c][3], e);
            o.z = pack2(v[2 * c + 1][0], v[2 * c + 1][1], e); o.w = pack2(v[2 * c + 1][2], v[2 * c + 1][3], e);
            __builtin_amdgcn_raw_buffer_store_b128(o, rs, ioff + lds_byte(row & 127, kl + 8 * c), 0, 17);
        }
        if ((tid & 255) == 0)
            __builtin_amdgcn_raw_buffer_store_b32((unsigned)e, rs, (unsigned)(OFF_EX + (call & 1) * EX_BYTES) + row * 4, 0, 17);
        asm volatile("s_waitcnt vmcnt(0)" ::: "memory");
        __syncthreads();
    }
    if (tid == 0) {
        const unsigned xticket = __hip_atomic_fetch_add((unsigned*)(ws + OFF_CTR_X), 1u, __ATOMIC_RELAXED, __HIP_MEMORY_SCOPE_AGENT);
        if (xticket % NWG == NWG - 1) {
            __builtin_amdgcn_fence(__ATOMIC_ACQUIRE, "agent");
            __builtin_amdgcn_fence(__ATOMIC_RELEASE, "");
            asm volatile("s_waitcnt vmcnt(0)" ::: "memory");
            const unsigned call = ctl[0];
#pragma unroll
            for (int r = 0; r < NR; ++r)
                __hip_atomic_store((unsigned*)(p.ws[r] + OFF_FLAG + 128 * p.rank), call, __ATOMIC_RELAXED, __HIP_MEMORY_SCOPE_SYSTEM);
        }
    }
    {
        const int n0 = ((b & 7) * 32 + (b >> 3)) * 16, q = tid & 3, g = tid >> 2;
        f32x4 v[4][8];
#pragma unroll
        for (int i = 0; i < 4; ++i)
#pragma unroll
            for (int j = 0; j < 8; ++j)
                v[i][j] = *(const f32x4*)(p.w + (size_t)((i * 128 + g) * 8 + j) * NLOC + n0 + 4 * q);
        float cm[4] = {0.f, 0.f, 0.f, 0.f};
#pragma unroll
        for (int i = 0; i < 4; ++i)
#pragma unroll
            for (int j = 0; j < 8; ++j)
#pragma unroll
                for (int c = 0; c < 4; ++c) cm[c] = __builtin_fmaxf(cm[c], __builtin_fabsf(v[i][j][c]));
#pragma unroll
        for (int c = 0; c < 4; ++c)
#pragma unroll
            for (int s = 4; s < 64; s <<= 1) cm[c] = __builtin_fmaxf(cm[c], __shfl_xor(cm[c], s));
        __syncthreads();
        if (lane < 4) {
#pragma unroll
            for (int c = 0; c < 4; ++c) red[wid * 16 + 4 * lane + c] = cm[c];
        }
        __syncthreads();
        int e[4];
#pragma unroll
        for (int c = 0; c < 4; ++c) {
            float m = 0.f;
#pragma unroll
            for (int w = 0; w < 8; ++w) m = __builtin_fmaxf(m, red[w * 16 + 4 * q + c]);
            e[c] = scale_exp(m);
        }
        char* w16 = ws + OFF_W16;
#pragma unroll
        for (int i = 0; i < 4; ++i) {
            const int k0 = (i * 128 + g) * 8, ks = k0 >> 6, kl = k0 & 63;
#pragma unroll
            for (int c = 0; c < 4; ++c) {
                const int n = n0 + 4 * q + c;
                u32x4 o;
                o.x = pack2(v[i][0][c], v[i][1][c], e[c]); o.y = pack2(v[i][2][c], v[i][3][c], e[c]);
                o.z = pack2(v[i][4][c], v[i][5][c], e[c]); o.w = pack2(v[i][6][c], v[i][7][c], e[c]);
                *(u32x4*)(w16 + (size_t)(((n >> 8) * NT + ks) * 2 + ((n >> 7) & 1)) * HTB + lds_byte(n & 127, kl)) = o;
            }
        }
        if (g == 0) {
#pragma unroll
            for (int c = 0; c < 4; ++c) ((int*)(ws + OFF_EW))[n0 + 4 * q + c] = e[c];
        }
    }
}

__device__ __forceinline__ void gemm_phase(const Params& p, LAS char* lds) {
    const int tid = threadIdx.x, wid = __builtin_amdgcn_readfirstlane(tid >> 6), lane = tid & 63;
    const int wr = wid >> 2, wc = wid & 3, fr = lane & 15, fq = lane >> 4;
    const int b = blockIdx.x, src = b & 7, idx = b >> 3, mt = idx & 1, pn = idx >> 1;
    char* ws = p.ws[p.rank];
    LAS unsigned* ctl = (LAS unsigned*)(lds + LDS_CTL);

#define STAMP(i)
    STAMP(0);
    if (wid == 0) {
        const unsigned call = __builtin_amdgcn_readfirstlane(
            __hip_atomic_load((unsigned*)(ws + OFF_CTR_CALL), __ATOMIC_RELAXED, __HIP_MEMORY_SCOPE_AGENT)) / NWG;
        unsigned* flag = (unsigned*)(ws + OFF_FLAG + 128 * src);
        unsigned* tmo = (unsigned*)(ws + OFF_TMO);
        const unsigned long long t0 = __builtin_amdgcn_s_memrealtime();
        while ((unsigned)__builtin_amdgcn_readfirstlane(__hip_atomic_load(flag, __ATOMIC_RELAXED, __HIP_MEMORY_SCOPE_SYSTEM)) < call) {
            if ((unsigned)__builtin_amdgcn_readfirstlane(__hip_atomic_load(tmo, __ATOMIC_RELAXED, __HIP_MEMORY_SCOPE_AGENT)) == call
                || __builtin_amdgcn_s_memrealtime() - t0 > DEADLINE_TICKS) {
                if (lane == 0) __hip_atomic_store(tmo, call, __ATOMIC_RELAXED, __HIP_MEMORY_SCOPE_AGENT);
                break;
            }
            __builtin_amdgcn_s_sleep(2);
        }
        __builtin_amdgcn_fence(__ATOMIC_ACQUIRE, "");
        asm volatile("s_waitcnt vmcnt(0)" ::: "memory");
        const unsigned dead = (unsigned)__builtin_amdgcn_readfirstlane(__hip_atomic_load(tmo, __ATOMIC_RELAXED, __HIP_MEMORY_SCOPE_AGENT)) == call;
        if (lane == 0) { ctl[0] = call; ctl[1] = dead; }
    }
    __syncthreads();
    STAMP(1);
    const unsigned call = __builtin_amdgcn_readfirstlane(ctl[0]);
    const bool dead = __builtin_amdgcn_readfirstlane(ctl[1]) != 0;
    float* outp = p.out + (size_t)(src * MLOC + mt * 256 + wr * 64 + fr) * NLOC + pn * 256 + wc * 32 + 4 * fq;
    if (dead) {
        const float q = __builtin_nanf("");
        for (int ai = 0; ai < 2; ++ai) for (int m = 0; m < 4; ++m) for (int bj = 0; bj < 2; ++bj) for (int n = 0; n < 2; ++n)
            *(f32x4*)(outp + (size_t)(ai * HALF + m * 16) * NLOC + bj * HALF + n * 16) = (f32x4){q, q, q, q};
        return;
    }
    const char* xs = p.ws[src] + OFF_X16 + (call & 1) * X16_BYTES;
    const char* cA = xs + mt * 2 * HTB + tid * 16;
    const char* cB = ws + OFF_W16 + (size_t)pn * (NT * 2 * HTB) + tid * 16;
    const char* cP = xs + (idx * 8 + wid) * 256 + lane * 4;
    const unsigned ldsw = (unsigned)wid * 1024u;
    const int aoff = lds_byte(wr * 64 + fr, fq * 8), boff = lds_byte(wc * 32 + fr, fq * 8);

#define SA(bf, h) (((bf) * 2 + (h)) * HTB)
#define SB(bf, h) ((4 + (bf) * 2 + (h)) * HTB)
#define STAGE(bufoff, gptr) do { _Pragma("unroll") for (int _i = 0; _i < 2; ++_i) \
        __builtin_amdgcn_global_load_lds((const unsigned*)((gptr) + _i * 8192), (LAS unsigned*)(lds + (bufoff) + ldsw + _i * 8192), 16, 0, 0); } while (0)
#define PREF(t) __builtin_amdgcn_global_load_lds((const unsigned*)(cP + (size_t)(t) * 65536), (LAS unsigned*)(lds + LDS_PF + wid * 256), 4, 0, 0)
#define LDA(dst, bf, h) do { _Pragma("unroll") for (int m = 0; m < 4; ++m) _Pragma("unroll") for (int k = 0; k < 2; ++k) \
        dst[m][k] = *(const LAS f16x8*)(lds + SA(bf, h) + aoff + m * 2048 + k * 1024); } while (0)
#define LDB(dst, bf, h) do { _Pragma("unroll") for (int n = 0; n < 2; ++n) _Pragma("unroll") for (int k = 0; k < 2; ++k) \
        dst[n][k] = *(const LAS f16x8*)(lds + SB(bf, h) + boff + n * 2048 + k * 1024); } while (0)
#define MMA(ai, bj, At, Bt) do { __builtin_amdgcn_s_setprio(1); _Pragma("unroll") for (int m = 0; m < 4; ++m) _Pragma("unroll") for (int n = 0; n < 2; ++n) \
        _Pragma("unroll") for (int k = 0; k < 2; ++k) \
        acc[ai][bj][m][n] = __builtin_amdgcn_mfma_f32_16x16x32_f16(Bt[n][k], At[m][k], acc[ai][bj][m][n], 0, 0, 0); __builtin_amdgcn_s_setprio(0); } while (0)
#define WAIT_V(n) asm volatile("s_waitcnt vmcnt(" #n ")" ::: "memory")
#define WAIT_L(n) asm volatile("s_waitcnt lgkmcnt(" #n ")" ::: "memory")
#define BAR __builtin_amdgcn_s_barrier()
#define SCHED __builtin_amdgcn_sched_barrier(0)

    f32x4 acc[2][2][4][2];
#pragma unroll
    for (int a = 0; a < 2; ++a)
#pragma unroll
        for (int c = 0; c < 2; ++c)
#pragma unroll
            for (int m = 0; m < 4; ++m)
#pragma unroll
                for (int n = 0; n < 2; ++n) acc[a][c][m][n] = (f32x4){0.f, 0.f, 0.f, 0.f};
    f16x8 At[4][2], B0[2][2], B1[2][2];

    {
        const char* esrc = wid < 4 ? (p.ws[src] + OFF_EX + (call & 1) * EX_BYTES + (mt * 256 + wid * 64 + lane) * 4)
                                   : (ws + OFF_EW + (pn * 256 + (wid - 4) * 64 + lane) * 4);
        __builtin_amdgcn_global_load_lds((const unsigned*)esrc, (LAS unsigned*)(lds + LDS_EXP + wid * 256), 4, 0, 0);
    }
#pragma unroll
    for (int t = 2; t < PF_DIST; ++t) PREF(t);
    STAGE(SB(0, 0), cB); STAGE(SA(0, 0), cA); STAGE(SB(0, 1), cB + HTB); STAGE(SA(0, 1), cA + HTB);
    if (wr == 1) BAR;
    WAIT_V(4); BAR;
    STAGE(SB(1, 0), cB + 2 * HTB); STAGE(SA(1, 0), cA + 4 * HTB); STAGE(SB(1, 1), cB + 3 * HTB);
    WAIT_V(6); BAR;
    STAMP(2);
    for (int t = 0; t < NT; t += 2) {
        const int t2 = t + 2 < NT ? t + 2 : NT - 2, t3 = t2 + 1;
        const int p0 = t + PF_DIST < NT ? t + PF_DIST : NT - 1, p1 = t + 1 + PF_DIST < NT ? t + 1 + PF_DIST : NT - 1;
        const char* a1 = cA + (size_t)(t + 1) * 65536;
        const char* a2 = cA + (size_t)t2 * 65536; const char* b2 = cB + (size_t)t2 * 32768;
        const char* a3 = cA + (size_t)t3 * 65536; const char* b3 = cB + (size_t)t3 * 32768;
        LDB(B0, 0, 0); SCHED; LDA(At, 0, 0); STAGE(SA(1, 1), a1 + HTB); PREF(p0);
        WAIT_L(8); BAR; WAIT_L(0); MMA(0, 0, At, B0); BAR; SCHED;
        LDB(B1, 0, 1); STAGE(SB(0, 0), b2);
        BAR; WAIT_L(0); MMA(0, 1, At, B1); BAR;
        LDA(At, 0, 1); STAGE(SA(0, 0), a2);
        BAR; WAIT_L(0); MMA(1, 0, At, B0); BAR; SCHED;
        STAGE(SB(0, 1), b2 + HTB);
        WAIT_V(7); BAR; MMA(1, 1, At, B1); BAR;
        LDB(B0, 1, 0); SCHED; LDA(At, 1, 0); STAGE(SA(0, 1), a2 + HTB); PREF(p1);
        WAIT_L(8); BAR; WAIT_L(0); MMA(0, 0, At, B0); BAR; SCHED;
        LDB(B1, 1, 1); STAGE(SB(1, 0), b3);
        BAR; WAIT_L(0); MMA(0, 1, At, B1); BAR;
        LDA(At, 1, 1); STAGE(SA(1, 0), a3);
        BAR; WAIT_L(0); MMA(1, 0, At, B0); BAR; SCHED;
        STAGE(SB(1, 1), b3 + HTB);
        WAIT_V(7); BAR; MMA(1, 1, At, B1); BAR;
    }
    STAMP(3);
    WAIT_V(0);
    if (wr == 0) BAR;
    const LAS int* ex = (const LAS int*)(lds + LDS_EXP);
    const LAS int* ew = ex + 256;
    u32x4 ce[2][2];
#pragma unroll
    for (int bj = 0; bj < 2; ++bj)
#pragma unroll
        for (int n = 0; n < 2; ++n) ce[bj][n] = *(const LAS u32x4*)(ew + bj * HALF + wc * 32 + n * 16 + 4 * fq);
#pragma unroll
    for (int ai = 0; ai < 2; ++ai)
#pragma unroll
        for (int m = 0; m < 4; ++m) {
            const int re = ex[ai * HALF + wr * 64 + m * 16 + fr];
            float* rowp = outp + (size_t)(ai * HALF + m * 16) * NLOC;
#pragma unroll
            for (int bj = 0; bj < 2; ++bj)
#pragma unroll
                for (int n = 0; n < 2; ++n) {
                    f32x4 o;
#pragma unroll
                    for (int j = 0; j < 4; ++j)
                    o[j] = silu(__builtin_amdgcn_ldexpf(acc[ai][bj][m][n][j], re + (int)ce[bj][n][j]));
                    *(f32x4*)(rowp + bj * HALF + n * 16) = o;
                }
        }
}

__global__ void __launch_bounds__(512, 2) ag_gemm_silu(Params p, int phase) {
    extern __shared__ __attribute__((aligned(16))) char smem[];
    LAS char* lds = (LAS char*)smem;
    if (phase == 0) prep_phase(p, lds);
    else gemm_phase(p, lds);
}

extern "C" void kernel_launch(int rank, int nranks, ncclComm_t comm,
                              hipStream_t stream,
                              void* const* d_in, const int* in_sizes, int n_in,
                              void* d_out, int out_size,
                              const int* in_dtypes, int out_dtype,
                              void* const* d_out_all,
                              void* const* d_ws_all, size_t ws_bytes) {
    if (nranks != NR || (unsigned)rank >= (unsigned)NR || ws_bytes < WS_NEEDED || n_in < 2
        || in_sizes[0] != MLOC * KD || in_sizes[1] != KD * NLOC || out_size != NR * MLOC * NLOC) {
        fprintf(stderr, "kernel_launch: unexpected problem shape\n");
        return;
    }
    static bool attr_set = false;
    if (!attr_set) {
        HIP_CHECK(hipFuncSetAttribute((const void*)ag_gemm_silu, hipFuncAttributeMaxDynamicSharedMemorySize, LDS_TOTAL));
        attr_set = true;
    }
    Params p;
    p.x = (const float*)d_in[0];
    p.w = (const float*)d_in[1];
    p.out = (float*)d_out;
    for (int q = 0; q < NR; ++q) p.ws[q] = (char*)d_ws_all[q];
    p.rank = rank;
    ag_gemm_silu<<<NWG, 512, 2048, stream>>>(p, 0);
    ag_gemm_silu<<<NWG, 512, LDS_TOTAL, stream>>>(p, 1);
    HIP_CHECK(hipGetLastError());
}
```

```cpp
#include <hip/hip_runtime.h>
#include <rccl/rccl.h>
#include <cstdio>
#include <cstdint>

#define HIP_CHECK(expr) do { hipError_t _e = (expr); if (_e != hipSuccess) { \
    fprintf(stderr, "HIP error %s at %s:%d\n", hipGetErrorString(_e), __FILE__, __LINE__); \
    return; } } while (0)

#define LAS __attribute__((address_space(3)))
typedef _Float16 f16x8 __attribute__((ext_vector_type(8)));
typedef float f32x4 __attribute__((ext_vector_type(4)));
typedef unsigned u32x4 __attribute__((ext_vector_type(4)));
typedef int i32x4 __attribute__((ext_vector_type(4)));

constexpr int NR = 8;
constexpr int KD = 4096;
constexpr int NLOC = 4096;
constexpr int MLOC = 512;
constexpr int BK = 64, HALF = 128;
constexpr int HTB = HALF * BK * 2;
constexpr int STAGE_BYTES = 8 * HTB;
constexpr int NT = KD / BK;
constexpr int NWG = 256;
constexpr int PF_DIST = 4;
constexpr int NG = 16;
constexpr int D = 4;

constexpr size_t OFF_CTR_CALL = 0, OFF_CTR_X = 256, OFF_TMO = 512, OFF_FLAG = 4096;
constexpr size_t OFF_CNT = 8192;
constexpr size_t OFF_X16 = 1u << 20, X16_BYTES = 4u << 20;
constexpr size_t OFF_EX = 9u << 20, EX_BYTES = 4096;
constexpr size_t OFF_W16 = 16u << 20;
constexpr size_t OFF_EW = 48u << 20;
constexpr size_t WS_NEEDED = 49u << 20;

constexpr int LDS_CV = STAGE_BYTES;
constexpr int LDS_EX = LDS_CV + 16384;
constexpr int LDS_EW = LDS_EX + 1024;
constexpr int LDS_PF = LDS_EW + 2048;
constexpr int LDS_POLL = LDS_PF + 2048;
constexpr int LDS_RED = LDS_POLL + 2048;
constexpr int LDS_CTL = LDS_RED + 512;
constexpr int LDS_TOTAL = LDS_CTL + 512;

constexpr unsigned long long DEADLINE_TICKS = 100000000ull;

struct Params {
    const float* x;
    const float* w;
    float* out;
    char* ws[NR];
    int rank;
};

__device__ __forceinline__ int lds_byte(int r, int c) {
    const int st = (r >> 4) * 2 + (c >> 5), rr = r & 15, cc = c & 31, ob = rr * 64 + cc * 2;
    return st * 1024 + (ob ^ (((ob >> 9) & 1) << 5));
}
__device__ __forceinline__ unsigned w16_off(int n, int k) {
    return (unsigned)OFF_W16 + (unsigned)(((n >> 8) * NT + (k >> 6)) * 2 + ((n >> 7) & 1)) * HTB + lds_byte(n & 127, k & 63);
}

__device__ __forceinline__ int scale_exp(float amax) {
    return amax > 0.f ? __builtin_amdgcn_frexp_expf(amax) - 15 : -200;
}

__device__ __forceinline__ int grow_exp(int erun, float amax) {
    const int e = scale_exp(amax);
    return e > erun ? e + 1 : erun;
}

__device__ __forceinline__ unsigned pack2(float a, float b, int e) {
    const _Float16 ha = (_Float16)__builtin_amdgcn_ldexpf(a, -e), hb = (_Float16)__builtin_amdgcn_ldexpf(b, -e);
    return (unsigned)__builtin_bit_cast(unsigned short, ha) | ((unsigned)__builtin_bit_cast(unsigned short, hb) << 16);
}

__device__ __forceinline__ float silu(float y) {
    const float c = __builtin_fminf(__builtin_fmaxf(y, -60.f), 60.f);
    const float e = __builtin_amdgcn_exp2f(c * -1.44269504088896341f);
    return y * __builtin_amdgcn_rcpf(1.0f + e);
}

template <bool SYS>
__device__ __forceinline__ bool wait_ge(unsigned* word, unsigned want, unsigned* tmo, unsigned call, int lane) {
    const unsigned long long t0 = __builtin_amdgcn_s_memrealtime();
    for (;;) {
        const unsigned v = SYS ? __hip_atomic_load(word, __ATOMIC_RELAXED, __HIP_MEMORY_SCOPE_SYSTEM)
                               : __hip_atomic_load(word, __ATOMIC_RELAXED, __HIP_MEMORY_SCOPE_AGENT);
        if ((unsigned)__builtin_amdgcn_readfirstlane(v) >= want) return true;
        if ((unsigned)__builtin_amdgcn_readfirstlane(__hip_atomic_load(tmo, __ATOMIC_RELAXED, __HIP_MEMORY_SCOPE_AGENT)) == call
            || __builtin_amdgcn_s_memrealtime() - t0 > DEADLINE_TICKS) {
            if (lane == 0) __hip_atomic_store(tmo, call, __ATOMIC_RELAXED, __HIP_MEMORY_SCOPE_AGENT);
            return false;
        }
        __builtin_amdgcn_s_sleep(2);
    }
}

__global__ void __launch_bounds__(512, 2) ag_gemm_silu(Params p) {
    extern __shared__ __attribute__((aligned(16))) char smem[];
    LAS char* lds = (LAS char*)smem;
    const int tid = threadIdx.x, wid = __builtin_amdgcn_readfirstlane(tid >> 6), lane = tid & 63;
    const int wr = wid >> 2, wc = wid & 3, fr = lane & 15, fq = lane >> 4;
    const int b = blockIdx.x, src = b & 7, idx = b >> 3, mt = idx & 1, pn = idx >> 1;
    char* ws = p.ws[p.rank];
    LAS unsigned* ctl = (LAS unsigned*)(lds + LDS_CTL);
    LAS float* red = (LAS float*)(lds + LDS_RED);
    unsigned* tmo = (unsigned*)(ws + OFF_TMO);
    const __amdgpu_buffer_rsrc_t rs = __builtin_amdgcn_make_buffer_rsrc(ws, 0, (int)WS_NEEDED, 0x00020000);
#define STAMP(i)
#define TB()
#define TE(i)
#define SST(i)
    STAMP(0);
    const int n0 = ((b & 7) * 32 + (b >> 3)) * 16;
    const char* wbase = (const char*)(p.w + n0);
    unsigned* mycnt = (unsigned*)(ws + OFF_CNT + (n0 >> 8) * 128);
    int erun;

    if (tid == 0) {
        const unsigned t = __hip_atomic_fetch_add((unsigned*)(ws + OFF_CTR_CALL), 1u, __ATOMIC_RELAXED, __HIP_MEMORY_SCOPE_AGENT);
        ctl[0] = t / NWG + 1;
    }
    if (tid == 64) __builtin_amdgcn_fence(__ATOMIC_ACQUIRE, "");
    unsigned call;
    {
        const int row = 2 * b + (tid >> 8), kk = (tid & 255) * 16, cq = tid & 3, cg = tid >> 2;
        const f32x4* xsrc = (const f32x4*)(p.x + (size_t)row * KD + kk);
        f32x4 v[4], wv[8];
#pragma unroll
        for (int i = 0; i < 4; ++i) v[i] = xsrc[i];
#pragma unroll
        for (int j = 0; j < 8; ++j) wv[j] = *(const f32x4*)(wbase + (size_t)(cg * 8 + j) * (NLOC * 4) + cq * 16);
        float m = 0.f;
#pragma unroll
        for (int i = 0; i < 4; ++i)
#pragma unroll
            for (int j = 0; j < 4; ++j) m = __builtin_fmaxf(m, __builtin_fabsf(v[i][j]));
#pragma unroll
        for (int s = 1; s < 64; s <<= 1) m = __builtin_fmaxf(m, __shfl_xor(m, s));
        float cm[4] = {0.f, 0.f, 0.f, 0.f};
#pragma unroll
        for (int j = 0; j < 8; ++j)
#pragma unroll
            for (int c = 0; c < 4; ++c) cm[c] = __builtin_fmaxf(cm[c], __builtin_fabsf(wv[j][c]));
#pragma unroll
        for (int c = 0; c < 4; ++c)
#pragma unroll
            for (int s = 4; s < 64; s <<= 1) cm[c] = __builtin_fmaxf(cm[c], __shfl_xor(cm[c], s));
        LAS float* redx = (LAS float*)(lds + LDS_CTL + 64);
        if (lane == 0) redx[wid] = m;
        if (lane < 4) {
#pragma unroll
            for (int c = 0; c < 4; ++c) red[(4 * lane + c) * 8 + wid] = cm[c];
        }
        __syncthreads();
        SST(0);
        const int w0 = (tid >> 8) * 4;
        m = __builtin_fmaxf(__builtin_fmaxf(redx[w0], redx[w0 + 1]), __builtin_fmaxf(redx[w0 + 2], redx[w0 + 3]));
        call = __builtin_amdgcn_readfirstlane(ctl[0]);
        const int e = scale_exp(m);
        const unsigned ioff = (unsigned)(OFF_X16 + (call & 1) * X16_BYTES) + (unsigned)(kk >> 6) * 65536u + (unsigned)(row >> 7) * HTB;
        const int kl = kk & 63;
#pragma unroll
        for (int c = 0; c < 2; ++c) {
            u32x4 o;
            o.x = pack2(v[2 * c][0], v[2 * c][1], e); o.y = pack2(v[2 * c][2], v[2 * c][3], e);
            o.z = pack2(v[2 * c + 1][0], v[2 * c + 1][1], e); o.w = pack2(v[2 * c + 1][2], v[2 * c + 1][3], e);
            __builtin_amdgcn_raw_buffer_store_b128(o, rs, ioff + lds_byte(row & 127, kl + 8 * c), 0, 17);
        }
        if ((tid & 255) == 0)
            __builtin_amdgcn_raw_buffer_store_b32((unsigned)e, rs, (unsigned)(OFF_EX + (call & 1) * EX_BYTES) + row * 4, 0, 17);
        int e4[4];
#pragma unroll
        for (int c = 0; c < 4; ++c) {
            float mm = 0.f;
#pragma unroll
            for (int w = 0; w < 8; ++w) mm = __builtin_fmaxf(mm, red[(4 * cq + c) * 8 + w]);
            e4[c] = grow_exp(-300, mm);
        }
        {
            float mm = 0.f;
#pragma unroll
            for (int w = 0; w < 8; ++w) mm = __builtin_fmaxf(mm, red[(tid & 15) * 8 + w]);
            erun = grow_exp(-300, mm);
        }
        const unsigned woff = w16_off(n0 + 4 * cq, cg * 8);
#pragma unroll
        for (int c = 0; c < 4; ++c) {
            u32x4 o;
            o.x = pack2(wv[0][c], wv[1][c], e4[c]); o.y = pack2(wv[2][c], wv[3][c], e4[c]);
            o.z = pack2(wv[4][c], wv[5][c], e4[c]); o.w = pack2(wv[6][c], wv[7][c], e4[c]);
            __builtin_amdgcn_raw_buffer_store_b128(o, rs, woff + c * 64, 0, 16);
        }
        if (tid < 16) __builtin_amdgcn_raw_buffer_store_b32((unsigned)erun, rs, (unsigned)OFF_EW + (D - 1) * (NLOC * 4) + (n0 + tid) * 4, 0, 16);
        asm volatile("s_waitcnt vmcnt(0)" ::: "memory");
        __syncthreads();
        SST(1);
        if (tid == 64) __hip_atomic_fetch_add(mycnt + (D - 1) * 512, 1u, __ATOMIC_RELAXED, __HIP_MEMORY_SCOPE_AGENT);
        if (tid == 0) {
            const unsigned xticket = __hip_atomic_fetch_add((unsigned*)(ws + OFF_CTR_X), 1u, __ATOMIC_RELAXED, __HIP_MEMORY_SCOPE_AGENT);
            if (xticket % NWG == NWG - 1) {
                __builtin_amdgcn_fence(__ATOMIC_ACQUIRE, "agent");
                __builtin_amdgcn_fence(__ATOMIC_RELEASE, "");
                asm volatile("s_waitcnt vmcnt(0)" ::: "memory");
#pragma unroll
                for (int r = 0; r < NR; ++r)
                    __hip_atomic_store((unsigned*)(p.ws[r] + OFF_FLAG + 128 * p.rank), call, __ATOMIC_RELAXED, __HIP_MEMORY_SCOPE_SYSTEM);
            }
        }
    }
    unsigned* cntp = (unsigned*)(ws + OFF_CNT + pn * 128);
    const unsigned cnt_want = 16u * call;
    bool dead = !wait_ge<false>(cntp + (D - 1) * 512, cnt_want, tmo, call, lane);
    SST(2);
    const char* cB = ws + OFF_W16 + (size_t)pn * (NT * 2 * HTB);
    const unsigned toff = (unsigned)tid * 16u, poff = (unsigned)lane * 4u;
    const unsigned ldsw = (unsigned)wid * 1024u;
#define SA(bf, h) (((bf) * 2 + (h)) * HTB)
#define SB(bf, h) ((4 + (bf) * 2 + (h)) * HTB)
#define STAGE(bufoff, gptr) do { _Pragma("unroll") for (int _i = 0; _i < 2; ++_i) \
        __builtin_amdgcn_global_load_lds((const unsigned*)((gptr) + _i * 8192 + toff), (LAS unsigned*)(lds + (bufoff) + ldsw + _i * 8192), 16, 0, 0); } while (0)
    STAGE(SB(0, 0), cB); STAGE(SB(0, 1), cB + HTB); STAGE(SB(1, 0), cB + 2 * HTB); STAGE(SB(1, 1), cB + 3 * HTB);
    dead = !wait_ge<true>((unsigned*)(ws + OFF_FLAG + 128 * src), call, tmo, call, lane) || dead;
    STAMP(1);
    const char* xs = p.ws[dead ? p.rank : src] + OFF_X16 + (call & 1) * X16_BYTES;
    const char* cA = xs + mt * 2 * HTB;
    const char* cP = xs + (idx * 8 + wid) * 256;
    const int aoff = lds_byte(wr * 64 + fr, fq * 8), boff = lds_byte(wc * 32 + fr, fq * 8);

#define PREF(t) __builtin_amdgcn_global_load_lds((const unsigned*)(cP + (size_t)(t) * 65536 + poff), (LAS unsigned*)(lds + LDS_PF + wid * 256), 4, 0, 0)
#define LDA(dst, bf, h) do { _Pragma("unroll") for (int m = 0; m < 4; ++m) _Pragma("unroll") for (int k = 0; k < 2; ++k) \
        dst[m][k] = *(const LAS f16x8*)(lds + SA(bf, h) + aoff + m * 2048 + k * 1024); } while (0)
#define LDB(dst, bf, h) do { _Pragma("unroll") for (int n = 0; n < 2; ++n) _Pragma("unroll") for (int k = 0; k < 2; ++k) \
        dst[n][k] = *(const LAS f16x8*)(lds + SB(bf, h) + boff + n * 2048 + k * 1024); } while (0)
#define MMA(ai, bj, At, Bt) do { __builtin_amdgcn_s_setprio(1); _Pragma("unroll") for (int m = 0; m < 4; ++m) _Pragma("unroll") for (int n = 0; n < 2; ++n) \
        _Pragma("unroll") for (int k = 0; k < 2; ++k) \
        acc[ai][bj][m][n] = __builtin_amdgcn_mfma_f32_16x16x32_f16(Bt[n][k], At[m][k], acc[ai][bj][m][n], 0, 0, 0); __builtin_amdgcn_s_setprio(0); } while (0)
#define WAIT_V(n) asm volatile("s_waitcnt vmcnt(" #n ")" ::: "memory")
#define WAIT_L(n) asm volatile("s_waitcnt lgkmcnt(" #n ")" ::: "memory")
#define BAR __builtin_amdgcn_s_barrier()
#define SCHED __builtin_amdgcn_sched_barrier(0)

    f32x4 acc[2][2][4][2];
#pragma unroll
    for (int a = 0; a < 2; ++a)
#pragma unroll
        for (int c = 0; c < 2; ++c)
#pragma unroll
            for (int m = 0; m < 4; ++m)
#pragma unroll
                for (int n = 0; n < 2; ++n) acc[a][c][m][n] = (f32x4){0.f, 0.f, 0.f, 0.f};
    f16x8 At[4][2], B0[2][2], B1[2][2];
    const unsigned cstoff = w16_off(n0 + (tid & 15), (tid >> 4) * 8);
    const unsigned aldoff = (unsigned)(tid >> 2) * (NLOC * 4) + (tid & 3) * 16;
    const int cvoff = LDS_CV + (tid >> 4) * 512 + (tid & 15) * 4;
    const int redoff = LDS_RED + (tid & 15) * 32;
    float cv[8];
#pragma unroll
    for (int r = 0; r < 8; ++r) cv[r] = 0.f;

    {
        const char* esrc = wid < 4 ? (xs - (call & 1) * X16_BYTES - OFF_X16 + OFF_EX + (call & 1) * EX_BYTES + (mt * 256 + wid * 64) * 4)
                                   : (ws + OFF_EW + (D - 1) * (NLOC * 4) + (pn * 256 + (wid - 4) * 64) * 4);
        const int edst = wid < 4 ? LDS_EX + wid * 256 : LDS_EW + ((D - 1) & 1) * 1024 + (wid - 4) * 256;
        __builtin_amdgcn_global_load_lds((const unsigned*)(esrc + poff), (LAS unsigned*)(lds + edst), 4, 0, 0);
    }
#pragma unroll
    for (int t = 2; t < PF_DIST; ++t) PREF(t);
    STAGE(SA(0, 0), cA); STAGE(SA(0, 1), cA + HTB); STAGE(SA(1, 0), cA + 4 * HTB);
    if (wr == 1) BAR;
    WAIT_V(2); BAR; BAR;
    STAMP(2);
    for (int T = 0; T < NT / 2; ++T) {
        const int t = 2 * T, odd = T & 1, jh = T >> 1;
        const int t2 = t + 2 < NT ? t + 2 : NT - 2, t3 = t2 + 1;
        const int p0 = t + PF_DIST < NT ? t + PF_DIST : NT - 1, p1 = t + 1 + PF_DIST < NT ? t + 1 + PF_DIST : NT - 1;
        const char* a1 = cA + (size_t)(t + 1) * 65536;
        const char* a2 = cA + (size_t)t2 * 65536; const char* b2 = cB + (size_t)t2 * 32768;
        const char* a3 = cA + (size_t)t3 * 65536; const char* b3 = cB + (size_t)t3 * 32768;
        TB();
        if (!odd && jh >= D) {
            const LAS int* eo = (const LAS int*)(lds + LDS_EW + ((jh - 1) & 1) * 1024) + wc * 32 + 4 * fq;
            const LAS int* en = (const LAS int*)(lds + LDS_EW + (jh & 1) * 1024) + wc * 32 + 4 * fq;
#pragma unroll
            for (int bj = 0; bj < 2; ++bj)
#pragma unroll
                for (int n = 0; n < 2; ++n) {
                    const i32x4 d = *(const LAS i32x4*)(eo + bj * HALF + n * 16) - *(const LAS i32x4*)(en + bj * HALF + n * 16);
                    if (__any((d[0] | d[1] | d[2] | d[3]) != 0)) {
#pragma unroll
                        for (int ai = 0; ai < 2; ++ai)
#pragma unroll
                            for (int m = 0; m < 4; ++m)
#pragma unroll
                                for (int j = 0; j < 4; ++j) acc[ai][bj][m][n][j] = __builtin_amdgcn_ldexpf(acc[ai][bj][m][n][j], d[j]);
                    }
                }
        }
        const bool need1 = jh + 1 >= D && jh + 1 < NG;
        const bool do_c = jh >= 1 && jh + D - 1 < NG;
        if (!odd && need1) {
            const unsigned pollv = *(volatile LAS unsigned*)(lds + LDS_POLL + wid * 256);
            if ((unsigned)__builtin_amdgcn_readfirstlane(pollv) < cnt_want)
                dead = !wait_ge<false>(cntp + (jh + 1) * 512, cnt_want, tmo, call, lane) || dead;
        }
        TE(0);
        LDB(B0, 0, 0); SCHED; LDA(At, 0, 0); STAGE(SA(1, 1), a1 + HTB); PREF(p0);
        WAIT_L(8); BAR; WAIT_L(0); MMA(0, 0, At, B0); BAR; SCHED;
        LDB(B1, 0, 1); STAGE(SB(0, 0), b2);
        BAR; WAIT_L(0); MMA(0, 1, At, B1); BAR;
        LDA(At, 0, 1); STAGE(SA(0, 0), a2);
        BAR; WAIT_L(0); MMA(1, 0, At, B0); BAR; SCHED;
        STAGE(SB(0, 1), b2 + HTB);
        TB(); WAIT_V(7); TE(1); BAR; MMA(1, 1, At, B1); BAR;
        TB();
        STAGE(SA(0, 1), a2 + HTB);
        SCHED;
        PREF(p1);
        {
            const int ja = jh + D;
            if (!odd) {
                { const int jp = jh + 2 < NG ? jh + 2 : NG - 1;
                  __builtin_amdgcn_global_load_lds((const unsigned*)(cntp + jp * 512), (LAS unsigned*)(lds + LDS_POLL + wid * 256), 4, 0, 16); }
                { const int je = jh + 1 < NG ? jh + 1 : NG - 1;
                  __builtin_amdgcn_global_load_lds((const unsigned*)(ws + OFF_EW + je * (NLOC * 4) + (pn * 256 + (wid & 3) * 64) * 4 + poff),
                                                   (LAS unsigned*)(lds + (wid < 4 && need1 ? LDS_EW + (je & 1) * 1024 + wid * 256 : LDS_PF + wid * 256)), 4, 0, 0); }
                {
                    const int jl = ja < NG ? ja : NG - 1;
                    const char* ab = wbase + (size_t)(jl * 256) * (NLOC * 4);
#pragma unroll
                    for (int i = 0; i < 2; ++i)
                        __builtin_amdgcn_global_load_lds((const unsigned*)(ab + (size_t)i * (128 * NLOC * 4) + aldoff), (LAS unsigned*)(lds + LDS_CV + i * 8192 + ldsw), 16, 0, 0);
                }
                if (do_c) {
                    const int jc = ja - 1;
                    const f32x4 r0 = *(const LAS f32x4*)(lds + redoff), r1 = *(const LAS f32x4*)(lds + redoff + 16);
                    const float mm = __builtin_fmaxf(__builtin_fmaxf(__builtin_fmaxf(r0[0], r0[1]), __builtin_fmaxf(r0[2], r0[3])),
                                                     __builtin_fmaxf(__builtin_fmaxf(r1[0], r1[1]), __builtin_fmaxf(r1[2], r1[3])));
                    erun = grow_exp(erun, mm);
                    u32x4 o;
                    o.x = pack2(cv[0], cv[1], erun); o.y = pack2(cv[2], cv[3], erun); o.z = pack2(cv[4], cv[5], erun); o.w = pack2(cv[6], cv[7], erun);
                    __builtin_amdgcn_raw_buffer_store_b128(o, rs, cstoff, jc * (4 * 2 * HTB), 16);
                    if (wid == 0) { if (lane < 16) __builtin_amdgcn_raw_buffer_store_b32((unsigned)erun, rs, poff, (unsigned)OFF_EW + jc * (NLOC * 4) + n0 * 4, 16); }
                }
            } else {
                if (wid == 7 && do_c) { if (lane == 0) __hip_atomic_fetch_add(mycnt + (ja - 1) * 512, 1u, __ATOMIC_RELAXED, __HIP_MEMORY_SCOPE_AGENT); }
                const LAS float* cvp = (const LAS float*)(lds + cvoff);
                float mm = 0.f;
#pragma unroll
                for (int r = 0; r < 8; ++r) { cv[r] = cvp[r * 16]; mm = __builtin_fmaxf(mm, __builtin_fabsf(cv[r])); }
                { const auto sw = __builtin_amdgcn_permlane16_swap(__builtin_bit_cast(unsigned, mm), __builtin_bit_cast(unsigned, mm), false, false);
                  mm = __builtin_fmaxf(__builtin_bit_cast(float, sw[0]), __builtin_bit_cast(float, sw[1])); }
                { const auto sw = __builtin_amdgcn_permlane32_swap(__builtin_bit_cast(unsigned, mm), __builtin_bit_cast(unsigned, mm), false, false);
                  mm = __builtin_fmaxf(__builtin_bit_cast(float, sw[0]), __builtin_bit_cast(float, sw[1])); }
                if (lane < 16) *(LAS float*)(lds + redoff + wid * 4) = mm;
            }
        }
        SCHED;
        TE(2 + odd);
        LDB(B0, 1, 0); SCHED; LDA(At, 1, 0);
        WAIT_L(8); BAR; WAIT_L(0); MMA(0, 0, At, B0); BAR; SCHED;
        LDB(B1, 1, 1); STAGE(SB(1, 0), b3);
        BAR; WAIT_L(0); MMA(0, 1, At, B1); BAR;
        LDA(At, 1, 1); STAGE(SA(1, 0), a3);
        BAR; WAIT_L(0); MMA(1, 0, At, B0); BAR; SCHED;
        STAGE(SB(1, 1), b3 + HTB);
        TB();
        if (!odd) {
            if (!do_c) WAIT_V(11); else if (wid == 0) WAIT_V(13); else WAIT_V(12);
        }
        else if (wid == 7 && do_c) WAIT_V(8);
        else WAIT_V(7);
        TE(4 + odd);
        BAR; MMA(1, 1, At, B1); BAR;
    }
    STAMP(3);
    WAIT_V(0);
    if (wr == 0) BAR;
    dead = dead || (unsigned)__builtin_amdgcn_readfirstlane(__hip_atomic_load(tmo, __ATOMIC_RELAXED, __HIP_MEMORY_SCOPE_AGENT)) == call;
    int tz = threadIdx.x; asm volatile("" : "+v"(tz));
    const int efr = tz & 15, efq = (tz >> 4) & 3;
    float* outp = p.out + (size_t)(src * MLOC + mt * 256 + wr * 64 + efr) * NLOC + pn * 256 + wc * 32 + 4 * efq;
    const LAS int* ex = (const LAS int*)(lds + LDS_EX);
    const LAS int* ew = (const LAS int*)(lds + LDS_EW + ((NG - 1) & 1) * 1024);
    i32x4 ce[2][2];
#pragma unroll
    for (int bj = 0; bj < 2; ++bj)
#pragma unroll
        for (int n = 0; n < 2; ++n) ce[bj][n] = *(const LAS i32x4*)(ew + bj * HALF + wc * 32 + n * 16 + 4 * efq);
    if (dead) {
        const float qnan = __builtin_nanf("");
        for (int r = 0; r < 8; ++r) for (int c = 0; c < 4; ++c)
            *(f32x4*)(outp + (size_t)((r >> 2) * HALF + (r & 3) * 16) * NLOC + (c >> 1) * HALF + (c & 1) * 16) = (f32x4){qnan, qnan, qnan, qnan};
        return;
    }
#pragma unroll
    for (int ai = 0; ai < 2; ++ai)
#pragma unroll
        for (int m = 0; m < 4; ++m) {
            const int re = ex[ai * HALF + wr * 64 + m * 16 + efr];
            float* rowp = outp + (size_t)(ai * HALF + m * 16) * NLOC;
#pragma unroll
            for (int bj = 0; bj < 2; ++bj)
#pragma unroll
                for (int n = 0; n < 2; ++n) {
                    f32x4 o;
#pragma unroll
                    for (int j = 0; j < 4; ++j) o[j] = silu(__builtin_amdgcn_ldexpf(acc[ai][bj][m][n][j], re + ce[bj][n][j]));
                    *(f32x4*)(rowp + bj * HALF + n * 16) = o;
                }
        }
}

extern "C" void kernel_launch(int rank, int nranks, ncclComm_t comm,
                              hipStream_t stream,
                              void* const* d_in, const int* in_sizes, int n_in,
                              void* d_out, int out_size,
                              const int* in_dtypes, int out_dtype,
                              void* const* d_out_all,
                              void* const* d_ws_all, size_t ws_bytes) {
    if (nranks != NR || (unsigned)rank >= (unsigned)NR || ws_bytes < WS_NEEDED || n_in < 2
        || in_sizes[0] != MLOC * KD || in_sizes[1] != KD * NLOC || out_size != NR * MLOC * NLOC) {
        fprintf(stderr, "kernel_launch: unexpected problem shape\n");
        return;
    }
    static bool attr_set = false;
    if (!attr_set) {
        HIP_CHECK(hipFuncSetAttribute((const void*)ag_gemm_silu, hipFuncAttributeMaxDynamicSharedMemorySize, LDS_TOTAL));
        attr_set = true;
    }
    Params p;
    p.x = (const float*)d_in[0];
    p.w = (const float*)d_in[1];
    p.out = (float*)d_out;
    for (int q = 0; q < NR; ++q) p.ws[q] = (char*)d_ws_all[q];
    p.rank = rank;
    ag_gemm_silu<<<NWG, 512, LDS_TOTAL, stream>>>(p);
    HIP_CHECK(hipGetLastError());
}
```

```cpp
#include <hip/hip_runtime.h>
#include <rccl/rccl.h>
#include <cstdio>
#include <cstdint>

#define HIP_CHECK(expr) do { hipError_t _e = (expr); if (_e != hipSuccess) { \
    fprintf(stderr, "HIP error %s at %s:%d\n", hipGetErrorString(_e), __FILE__, __LINE__); \
    return; } } while (0)

#define LAS __attribute__((address_space(3)))
typedef _Float16 f16x8 __attribute__((ext_vector_type(8)));
typedef float f32x4 __attribute__((ext_vector_type(4)));
typedef unsigned u32x4 __attribute__((ext_vector_type(4)));
typedef int i32x4 __attribute__((ext_vector_type(4)));
typedef float f32x2 __attribute__((ext_vector_type(2)));

constexpr int NR = 8;
constexpr int KD = 4096;
constexpr int NLOC = 4096;
constexpr int MLOC = 512;
constexpr int BK = 64, HALF = 128;
constexpr int HTB = HALF * BK * 2;
constexpr int STAGE_BYTES = 8 * HTB;
constexpr int NT = KD / BK;
constexpr int NWG = 256;
constexpr int PF_DIST = 4;
constexpr int NG = 16;
constexpr int D = 4;

constexpr size_t OFF_CTR_CALL = 0, OFF_CTR_X = 256, OFF_TMO = 512, OFF_FLAG = 4096;
constexpr size_t OFF_CNT = 8192;
constexpr int XR = 8;
constexpr size_t OFF_EX = 512u << 10, EX_BYTES = 4096;
constexpr size_t OFF_W16 = 1u << 20;
constexpr size_t OFF_EW = 33u << 20;
constexpr size_t OFF_X16 = 34u << 20, X16_BYTES = 4u << 20;
constexpr size_t WS_NEEDED = OFF_X16 + XR * X16_BYTES;

constexpr int LDS_CV = STAGE_BYTES;
constexpr int LDS_EX = LDS_CV + 16384;
constexpr int LDS_EW = LDS_EX + 1024;
constexpr int LDS_PF = LDS_EW + 2048;
constexpr int LDS_POLL = LDS_PF + 2048;
constexpr int LDS_RED = LDS_POLL + 2048;
constexpr int LDS_CTL = LDS_RED + 512;
constexpr int LDS_TOTAL = LDS_CTL + 512;

constexpr unsigned long long DEADLINE_TICKS = 100000000ull;

struct Params {
    const float* x;
    const float* w;
    float* out;
    char* ws[NR];
    int rank;
};

__device__ __forceinline__ int lds_byte(int r, int c) {
    const int st = (r >> 4) * 2 + (c >> 5), rr = r & 15, cc = c & 31, ob = rr * 64 + cc * 2;
    return st * 1024 + (ob ^ (((ob >> 9) & 1) << 5));
}
__device__ __forceinline__ unsigned w16_off(int n, int k) {
    return (unsigned)OFF_W16 + (unsigned)(((n >> 8) * NT + (k >> 6)) * 2 + ((n >> 7) & 1)) * HTB + lds_byte(n & 127, k & 63);
}

__device__ __forceinline__ int scale_exp(float amax) {
    return amax > 0.f ? __builtin_amdgcn_frexp_expf(amax) - 15 : -200;
}

__device__ __forceinline__ int grow_exp(int erun, float amax) {
    const int e = scale_exp(amax);
    return e > erun ? e + 1 : erun;
}

__device__ __forceinline__ unsigned pack2(float a, float b, int e) {
    const _Float16 ha = (_Float16)__builtin_amdgcn_ldexpf(a, -e), hb = (_Float16)__builtin_amdgcn_ldexpf(b, -e);
    return (unsigned)__builtin_bit_cast(unsigned short, ha) | ((unsigned)__builtin_bit_cast(unsigned short, hb) << 16);
}

__device__ __forceinline__ float silu(float y) {
    const float c = __builtin_fminf(__builtin_fmaxf(y, -60.f), 60.f);
    const float e = __builtin_amdgcn_exp2f(c * -1.44269504088896341f);
    return y * __builtin_amdgcn_rcpf(1.0f + e);
}

template <bool SYS>
__device__ __forceinline__ bool wait_ge(unsigned* word, unsigned want, unsigned* tmo, unsigned call, int lane) {
    const unsigned long long t0 = __builtin_amdgcn_s_memrealtime();
    for (;;) {
        const unsigned v = SYS ? __hip_atomic_load(word, __ATOMIC_RELAXED, __HIP_MEMORY_SCOPE_SYSTEM)
                               : __hip_atomic_load(word, __ATOMIC_RELAXED, __HIP_MEMORY_SCOPE_AGENT);
        if ((unsigned)__builtin_amdgcn_readfirstlane(v) >= want) return true;
        if ((unsigned)__builtin_amdgcn_readfirstlane(__hip_atomic_load(tmo, __ATOMIC_RELAXED, __HIP_MEMORY_SCOPE_AGENT)) == call
            || __builtin_amdgcn_s_memrealtime() - t0 > DEADLINE_TICKS) {
            if (lane == 0) __hip_atomic_store(tmo, call, __ATOMIC_RELAXED, __HIP_MEMORY_SCOPE_AGENT);
            return false;
        }
        __builtin_amdgcn_s_sleep(2);
    }
}

__global__ void __launch_bounds__(512, 2) ag_gemm_silu(Params p) {
    extern __shared__ __attribute__((aligned(16))) char smem[];
    LAS char* lds = (LAS char*)smem;
    const int tid = threadIdx.x, wid = __builtin_amdgcn_readfirstlane(tid >> 6), lane = tid & 63;
    const int wr = wid >> 2, wc = wid & 3, fr = lane & 15, fq = lane >> 4;
    const int b = blockIdx.x, src = b & 7, idx = b >> 3, mt = idx & 1, pn = idx >> 1;
    char* ws = p.ws[p.rank];
    LAS unsigned* ctl = (LAS unsigned*)(lds + LDS_CTL);
    LAS float* red = (LAS float*)(lds + LDS_RED);
    unsigned* tmo = (unsigned*)(ws + OFF_TMO);
    const __amdgpu_buffer_rsrc_t rs = __builtin_amdgcn_make_buffer_rsrc(ws, 0, (int)WS_NEEDED, 0x00020000);
#define STAMP(i)
#define TB()
#define TE(i)
#define SST(i)
    STAMP(0);
    const int n0 = ((b & 7) * 32 + (b >> 3)) * 16;
    const char* wbase = (const char*)(p.w + n0);
    unsigned* mycnt = (unsigned*)(ws + OFF_CNT + (n0 >> 8) * 128);
    int erun;

    unsigned call;
    {
        const int row = 2 * b + (tid >> 8), kk = (tid & 255) * 16, cq = tid & 3, cg = tid >> 2;
        const f32x4* xsrc = (const f32x4*)(p.x + (size_t)row * KD + kk);
        f32x4 v[4], wv[8];
#pragma unroll
        for (int i = 0; i < 4; ++i) v[i] = xsrc[i];
        asm volatile("" ::: "memory");
        if (tid == 0) {
            const unsigned t = __hip_atomic_fetch_add((unsigned*)(ws + OFF_CTR_CALL), 1u, __ATOMIC_RELAXED, __HIP_MEMORY_SCOPE_AGENT);
            ctl[0] = t / NWG + 1;
        }
        float m = 0.f;
#pragma unroll
        for (int i = 0; i < 4; ++i)
#pragma unroll
            for (int j = 0; j < 4; ++j) m = __builtin_fmaxf(m, __builtin_fabsf(v[i][j]));
#pragma unroll
        for (int s = 1; s < 64; s <<= 1) m = __builtin_fmaxf(m, __shfl_xor(m, s));
        LAS float* redx = (LAS float*)(lds + LDS_CTL + 64);
        if (lane == 0) redx[wid] = m;
        __syncthreads();
        const int w0 = (tid >> 8) * 4;
        m = __builtin_fmaxf(__builtin_fmaxf(redx[w0], redx[w0 + 1]), __builtin_fmaxf(redx[w0 + 2], redx[w0 + 3]));
        call = __builtin_amdgcn_readfirstlane(ctl[0]);
        const int e = scale_exp(m);
        const unsigned ioff = (unsigned)(OFF_X16 + (call % XR) * X16_BYTES) + (unsigned)(kk >> 6) * 65536u + (unsigned)(row >> 7) * HTB;
        const int kl = kk & 63;
#pragma unroll
        for (int c = 0; c < 2; ++c) {
            u32x4 o;
            o.x = pack2(v[2 * c][0], v[2 * c][1], e); o.y = pack2(v[2 * c][2], v[2 * c][3], e);
            o.z = pack2(v[2 * c + 1][0], v[2 * c + 1][1], e); o.w = pack2(v[2 * c + 1][2], v[2 * c + 1][3], e);
            __builtin_amdgcn_raw_buffer_store_b128(o, rs, ioff + lds_byte(row & 127, kl + 8 * c), 0, 17);
        }
        if ((tid & 255) == 0)
            __builtin_amdgcn_raw_buffer_store_b32((unsigned)e, rs, (unsigned)(OFF_EX + (call % XR) * EX_BYTES) + row * 4, 0, 17);
        asm volatile("" ::: "memory");
        if (wid != 0) {
#pragma unroll
            for (int j = 0; j < 8; ++j) wv[j] = *(const f32x4*)(wbase + (size_t)(cg * 8 + j) * (NLOC * 4) + cq * 16);
            asm volatile("s_waitcnt vmcnt(8)" ::: "memory");
        } else {
            asm volatile("s_waitcnt vmcnt(0)" ::: "memory");
        }
        __builtin_amdgcn_s_barrier();
        asm volatile("" ::: "memory");
        SST(0);
        if (wid == 0) {
            unsigned xticket = 0;
            if (tid == 0) xticket = __hip_atomic_fetch_add((unsigned*)(ws + OFF_CTR_X), 1u, __ATOMIC_RELAXED, __HIP_MEMORY_SCOPE_AGENT);
            asm volatile("" ::: "memory");
#pragma unroll
            for (int j = 0; j < 8; ++j) wv[j] = *(const f32x4*)(wbase + (size_t)(cg * 8 + j) * (NLOC * 4) + cq * 16);
            if (tid == 0 && xticket % NWG == NWG - 1) {
#pragma unroll
                for (int r = 0; r < NR; ++r)
                    __hip_atomic_store((unsigned*)(p.ws[r] + OFF_FLAG + 128 * p.rank), call, __ATOMIC_RELAXED, __HIP_MEMORY_SCOPE_SYSTEM);
            }
        }
        float cm[4] = {0.f, 0.f, 0.f, 0.f};
#pragma unroll
        for (int j = 0; j < 8; ++j)
#pragma unroll
            for (int c = 0; c < 4; ++c) cm[c] = __builtin_fmaxf(cm[c], __builtin_fabsf(wv[j][c]));
#pragma unroll
        for (int c = 0; c < 4; ++c)
#pragma unroll
            for (int s = 4; s < 64; s <<= 1) cm[c] = __builtin_fmaxf(cm[c], __shfl_xor(cm[c], s));
        if (lane < 4) {
#pragma unroll
            for (int c = 0; c < 4; ++c) red[(4 * lane + c) * 8 + wid] = cm[c];
        }
        __syncthreads();
        int e4[4];
#pragma unroll
        for (int c = 0; c < 4; ++c) {
            float mm = 0.f;
#pragma unroll
            for (int w = 0; w < 8; ++w) mm = __builtin_fmaxf(mm, red[(4 * cq + c) * 8 + w]);
            e4[c] = grow_exp(-300, mm);
        }
        {
            float mm = 0.f;
#pragma unroll
            for (int w = 0; w < 8; ++w) mm = __builtin_fmaxf(mm, red[(tid & 15) * 8 + w]);
            erun = grow_exp(-300, mm);
            if (tid < 16) *(LAS float*)(lds + LDS_CTL + 128 + tid * 4) = mm;
        }
        const unsigned woff = w16_off(n0 + 4 * cq, cg * 8);
#pragma unroll
        for (int c = 0; c < 4; ++c) {
            u32x4 o;
            o.x = pack2(wv[0][c], wv[1][c], e4[c]); o.y = pack2(wv[2][c], wv[3][c], e4[c]);
            o.z = pack2(wv[4][c], wv[5][c], e4[c]); o.w = pack2(wv[6][c], wv[7][c], e4[c]);
            __builtin_amdgcn_raw_buffer_store_b128(o, rs, woff + c * 64, 0, 16);
        }
        if (tid < 16) __builtin_amdgcn_raw_buffer_store_b32((unsigned)erun, rs, (unsigned)OFF_EW + (D - 1) * (NLOC * 4) + (n0 + tid) * 4, 0, 16);
        asm volatile("s_waitcnt vmcnt(0)" ::: "memory");
        __syncthreads();
        SST(1);
        if (tid == 64) __hip_atomic_fetch_add(mycnt + (D - 1) * 512, 1u, __ATOMIC_RELAXED, __HIP_MEMORY_SCOPE_AGENT);
    }
    int jnext = D;
    {
        unsigned* flag = (unsigned*)(ws + OFF_FLAG + 128 * src);
        for (; jnext < NG; ++jnext) {
            if (wid == 0) {
                unsigned ok = 0;
                for (int i = 0; i < 4 && !ok; ++i) {
                    ok = (unsigned)__builtin_amdgcn_readfirstlane(__hip_atomic_load(flag, __ATOMIC_RELAXED, __HIP_MEMORY_SCOPE_SYSTEM)) >= call;
                    if (!ok) __builtin_amdgcn_s_sleep(8);
                }
                if (lane == 0) ctl[2] = ok;
            }
            __syncthreads();
            if (__builtin_amdgcn_readfirstlane(ctl[2])) break;
            float cw[8];
#pragma unroll
            for (int r = 0; r < 8; ++r) cw[r] = *(const float*)(wbase + (size_t)(jnext * 256 + (tid >> 4) * 8 + r) * (NLOC * 4) + (tid & 15) * 4);
            float mm = 0.f;
#pragma unroll
            for (int r = 0; r < 8; ++r) mm = __builtin_fmaxf(mm, __builtin_fabsf(cw[r]));
            mm = __builtin_fmaxf(mm, __shfl_xor(mm, 16));
            mm = __builtin_fmaxf(mm, __shfl_xor(mm, 32));
            LAS unsigned* rmax = (LAS unsigned*)(lds + LDS_CTL + 128) + (tid & 15);
            if (lane < 16) __hip_atomic_fetch_max(rmax, __builtin_bit_cast(unsigned, mm), __ATOMIC_RELAXED, __HIP_MEMORY_SCOPE_WORKGROUP);
            __syncthreads();
            const int enew = grow_exp(erun, __builtin_bit_cast(float, *(volatile LAS unsigned*)rmax));
            u32x4 o;
            o.x = pack2(cw[0], cw[1], enew); o.y = pack2(cw[2], cw[3], enew); o.z = pack2(cw[4], cw[5], enew); o.w = pack2(cw[6], cw[7], enew);
            __builtin_amdgcn_raw_buffer_store_b128(o, rs, w16_off(n0 + (tid & 15), jnext * 256 + (tid >> 4) * 8), 0, 16);
            if (wid == 0) {
                if (tid < 16) __builtin_amdgcn_raw_buffer_store_b32((unsigned)enew, rs, tid * 4, (unsigned)OFF_EW + jnext * (NLOC * 4) + n0 * 4, 16);
                if (__any(enew != erun)) { if (tid == 0) __hip_atomic_store(mycnt + jnext * 512 + 1, call, __ATOMIC_RELAXED, __HIP_MEMORY_SCOPE_AGENT); }
            }
            erun = enew;
            asm volatile("s_waitcnt vmcnt(0)" ::: "memory");
            __syncthreads();
            if (tid == 64) __hip_atomic_fetch_add(mycnt + jnext * 512, 1u, __ATOMIC_RELAXED, __HIP_MEMORY_SCOPE_AGENT);
        }
    }
    unsigned* cntp = (unsigned*)(ws + OFF_CNT + pn * 128);
    const unsigned cnt_want = 16u * call;
    bool dead = !wait_ge<false>(cntp + (D - 1) * 512, cnt_want, tmo, call, lane);
    SST(2);
    const char* cB = ws + OFF_W16 + (size_t)pn * (NT * 2 * HTB);
    const unsigned toff = (unsigned)tid * 16u, poff = (unsigned)lane * 4u;
    const unsigned ldsw = (unsigned)wid * 1024u;
#define SA(bf, h) (((bf) * 2 + (h)) * HTB)
#define SB(bf, h) ((4 + (bf) * 2 + (h)) * HTB)
#define STAGE(bufoff, gptr) do { _Pragma("unroll") for (int _i = 0; _i < 2; ++_i) \
        __builtin_amdgcn_global_load_lds((const unsigned*)((gptr) + _i * 8192 + toff), (LAS unsigned*)(lds + (bufoff) + ldsw + _i * 8192), 16, 0, 0); } while (0)
    STAGE(SB(0, 0), cB); STAGE(SB(0, 1), cB + HTB); STAGE(SB(1, 0), cB + 2 * HTB); STAGE(SB(1, 1), cB + 3 * HTB);
    dead = !wait_ge<true>((unsigned*)(ws + OFF_FLAG + 128 * src), call, tmo, call, lane) || dead;
    STAMP(1);
    const char* xs = p.ws[dead ? p.rank : src] + OFF_X16 + (call % XR) * X16_BYTES;
    const char* cA = xs + mt * 2 * HTB;
    const char* cP = xs + (idx * 8 + wid) * 256;
    const int aoff = lds_byte(wr * 64 + fr, fq * 8), boff = lds_byte(wc * 32 + fr, fq * 8);

#define PREF(t) __builtin_amdgcn_global_load_lds((const unsigned*)(cP + (size_t)(t) * 65536 + poff), (LAS unsigned*)(lds + LDS_PF + wid * 256), 4, 0, 0)
#define LDA(dst, bf, h) do { _Pragma("unroll") for (int m = 0; m < 4; ++m) _Pragma("unroll") for (int k = 0; k < 2; ++k) \
        dst[m][k] = *(const LAS f16x8*)(lds + SA(bf, h) + aoff + m * 2048 + k * 1024); } while (0)
#define LDB(dst, bf, h) do { _Pragma("unroll") for (int n = 0; n < 2; ++n) _Pragma("unroll") for (int k = 0; k < 2; ++k) \
        dst[n][k] = *(const LAS f16x8*)(lds + SB(bf, h) + boff + n * 2048 + k * 1024); } while (0)
#define MMA(ai, bj, At, Bt) do { __builtin_amdgcn_s_setprio(1); _Pragma("unroll") for (int m = 0; m < 4; ++m) _Pragma("unroll") for (int n = 0; n < 2; ++n) \
        _Pragma("unroll") for (int k = 0; k < 2; ++k) \
        acc[ai][bj][m][n] = __builtin_amdgcn_mfma_f32_16x16x32_f16(Bt[n][k], At[m][k], acc[ai][bj][m][n], 0, 0, 0); __builtin_amdgcn_s_setprio(0); } while (0)
#define WAIT_V(n) asm volatile("s_waitcnt vmcnt(" #n ")" ::: "memory")
#define WAIT_L(n) asm volatile("s_waitcnt lgkmcnt(" #n ")" ::: "memory")
#define BAR __builtin_amdgcn_s_barrier()
#define SCHED __builtin_amdgcn_sched_barrier(0)

    f32x4 acc[2][2][4][2];
#pragma unroll
    for (int a = 0; a < 2; ++a)
#pragma unroll
        for (int c = 0; c < 2; ++c)
#pragma unroll
            for (int m = 0; m < 4; ++m)
#pragma unroll
                for (int n = 0; n < 2; ++n) acc[a][c][m][n] = (f32x4){0.f, 0.f, 0.f, 0.f};
    f16x8 At[4][2], B0[2][2], B1[2][2];
#define OPAQUE_TID(v) int v = threadIdx.x; asm volatile("" : "+v"(v))
    f32x2 cv[4];
#pragma unroll
    for (int r = 0; r < 4; ++r) cv[r] = (f32x2){0.f, 0.f};

    {
        if (wid < 4) __builtin_amdgcn_global_load_lds((const unsigned*)(xs - (call % XR) * X16_BYTES - OFF_X16 + OFF_EX + (call % XR) * EX_BYTES + (mt * 256 + wid * 64) * 4 + poff),
                                                      (LAS unsigned*)(lds + LDS_EX + wid * 256), 4, 0, 0);
    }
#pragma unroll
    for (int t = 2; t < PF_DIST; ++t) PREF(t);
    STAGE(SA(0, 0), cA); STAGE(SA(0, 1), cA + HTB); STAGE(SA(1, 0), cA + 4 * HTB);
    if (wr == 1) BAR;
    WAIT_V(2); BAR; BAR;
    STAMP(2);
#define MFMA16(ai, bj, At, Bt) do { _Pragma("unroll") for (int m = 0; m < 4; ++m) _Pragma("unroll") for (int n = 0; n < 2; ++n) \
        _Pragma("unroll") for (int k = 0; k < 2; ++k) \
        acc[ai][bj][m][n] = __builtin_amdgcn_mfma_f32_16x16x32_f16(Bt[n][k], At[m][k], acc[ai][bj][m][n], 0, 0, 0); } while (0)
#define INTERLEAVE(nv) do { _Pragma("unroll") for (int q = 0; q < 16; ++q) { __builtin_amdgcn_sched_group_barrier(0x8, 1, 0); __builtin_amdgcn_sched_group_barrier(0x2, nv, 0); } } while (0)
    bool grew = false;
    for (int jh = 0; jh < NG; ++jh) {
        const int t = 4 * jh;
        const bool need1 = jh + 1 >= D && jh + 1 < NG;
        const bool do_c = jh + D - 1 >= jnext && jh + D - 1 < NG;
        const int ja = jh + D < NG ? jh + D : NG - 1, jc = jh + D - 1;
        if (grew) {
            const int* eo = (const int*)(ws + OFF_EW + (jh - 1) * (NLOC * 4)) + pn * 256 + wc * 32 + 4 * fq;
#pragma unroll
            for (int bj = 0; bj < 2; ++bj)
#pragma unroll
                for (int n = 0; n < 2; ++n) {
                    i32x4 d;
#pragma unroll
                    for (int j = 0; j < 4; ++j)
                        d[j] = __hip_atomic_load(eo + bj * HALF + n * 16 + j, __ATOMIC_RELAXED, __HIP_MEMORY_SCOPE_AGENT)
                             - __hip_atomic_load(eo + NLOC + bj * HALF + n * 16 + j, __ATOMIC_RELAXED, __HIP_MEMORY_SCOPE_AGENT);
#pragma unroll
                    for (int ai = 0; ai < 2; ++ai)
#pragma unroll
                        for (int m = 0; m < 4; ++m)
#pragma unroll
                            for (int j = 0; j < 4; ++j) acc[ai][bj][m][n][j] = __builtin_amdgcn_ldexpf(acc[ai][bj][m][n][j], d[j]);
                }
            grew = false;
        }
        if (need1) {
            const unsigned pc = *(volatile LAS unsigned*)(lds + LDS_POLL + wid * 256), pg = *(volatile LAS unsigned*)(lds + LDS_POLL + wid * 256 + 4);
            unsigned chg = __builtin_amdgcn_readfirstlane(pg);
            if ((unsigned)__builtin_amdgcn_readfirstlane(pc) < cnt_want) {
                dead = !wait_ge<false>(cntp + (jh + 1) * 512, cnt_want, tmo, call, lane) || dead;
                chg = __builtin_amdgcn_readfirstlane(__hip_atomic_load(cntp + (jh + 1) * 512 + 1, __ATOMIC_RELAXED, __HIP_MEMORY_SCOPE_AGENT));
            }
            grew = chg == call;
        }
        {
            const char* a1 = cA + (size_t)(t + 1) * 65536;
            const char* a2 = cA + (size_t)(t + 2) * 65536; const char* b2 = cB + (size_t)(t + 2) * 32768;
            const char* a3 = cA + (size_t)(t + 3) * 65536; const char* b3 = cB + (size_t)(t + 3) * 32768;
            const int p0 = t + PF_DIST < NT ? t + PF_DIST : NT - 1, p1 = t + 1 + PF_DIST < NT ? t + 1 + PF_DIST : NT - 1;
            LDB(B0, 0, 0); SCHED; LDA(At, 0, 0); STAGE(SA(1, 1), a1 + HTB); PREF(p0);
            WAIT_L(8); BAR; WAIT_L(0); MMA(0, 0, At, B0); BAR; SCHED;
            LDB(B1, 0, 1); STAGE(SB(0, 0), b2);
            BAR; WAIT_L(0); MMA(0, 1, At, B1); BAR;
            LDA(At, 0, 1); STAGE(SA(0, 0), a2);
            float mm; { OPAQUE_TID(tq); mm = *(const LAS float*)(lds + LDS_CTL + 128 + (tq & 15) * 4); }
            BAR; WAIT_L(0); MMA(1, 0, At, B0); BAR; SCHED;
            STAGE(SB(0, 1), b2 + HTB);
            WAIT_V(7); BAR;
            u32x4 o; int enew; unsigned soff;
            __builtin_amdgcn_s_setprio(1);
            MFMA16(1, 1, At, B1);
            {
                enew = do_c ? grow_exp(erun, mm) : erun;
                o.x = pack2(cv[0][0], cv[0][1], enew); o.y = pack2(cv[1][0], cv[1][1], enew); o.z = pack2(cv[2][0], cv[2][1], enew); o.w = pack2(cv[3][0], cv[3][1], enew);
                OPAQUE_TID(tc);
                soff = w16_off(n0 + (tc & 15), (tc >> 4) * 8) | (do_c ? 0u : 0xFF000000u);
            }
            INTERLEAVE(3);
            asm volatile("" : "+v"(o), "+v"(enew), "+v"(soff));
            __builtin_amdgcn_s_setprio(0);
            BAR;
            STAGE(SA(0, 1), a2 + HTB);
            SCHED;
            PREF(p1);
            { const int jp = jh + 2 < NG ? jh + 2 : NG - 1;
              __builtin_amdgcn_global_load_lds((const unsigned*)((const char*)(cntp + jp * 512) + (poff & 4)), (LAS unsigned*)(lds + LDS_POLL + wid * 256), 4, 0, 16); }
            OPAQUE_TID(tq);
            { const char* ab = wbase + (size_t)(ja * 256) * (NLOC * 4);
              const unsigned aldoff = (unsigned)(tq >> 2) * (NLOC * 4) + (tq & 3) * 16;
#pragma unroll
              for (int i = 0; i < 2; ++i)
                  __builtin_amdgcn_global_load_lds((const unsigned*)(ab + (size_t)i * (128 * NLOC * 4) + aldoff), (LAS unsigned*)(lds + LDS_CV + i * 8192 + ldsw), 16, 0, 0); }
            __builtin_amdgcn_raw_buffer_store_b128(o, rs, soff, do_c ? jc * (4 * 2 * HTB) : 0, 16);
            if (wid == 0 && do_c) {
                if (tq < 16) __builtin_amdgcn_raw_buffer_store_b32((unsigned)enew, rs, tq * 4, (unsigned)OFF_EW + jc * (NLOC * 4) + n0 * 4, 16);
                if (__any(enew != erun)) { if (tq == 0) __hip_atomic_store(mycnt + jc * 512 + 1, call, __ATOMIC_RELAXED, __HIP_MEMORY_SCOPE_AGENT); }
            }
            erun = enew;
            SCHED;
            LDB(B0, 1, 0); SCHED; LDA(At, 1, 0);
            WAIT_L(8); BAR; WAIT_L(0); MMA(0, 0, At, B0); BAR; SCHED;
            LDB(B1, 1, 1); STAGE(SB(1, 0), b3);
            BAR; WAIT_L(0); MMA(0, 1, At, B1); BAR;
            LDA(At, 1, 1); STAGE(SA(1, 0), a3);
            BAR; WAIT_L(0); MMA(1, 0, At, B0); BAR; SCHED;
            STAGE(SB(1, 1), b3 + HTB);
            if (wid == 0 && do_c) WAIT_V(12); else WAIT_V(11);
            BAR; MMA(1, 1, At, B1); BAR;
        }
        {
            const int u2 = t + 4 < NT ? t + 4 : NT - 2, u3 = u2 + 1;
            const char* a1 = cA + (size_t)(t + 3) * 65536;
            const char* a2 = cA + (size_t)u2 * 65536; const char* b2 = cB + (size_t)u2 * 32768;
            const char* a3 = cA + (size_t)u3 * 65536; const char* b3 = cB + (size_t)u3 * 32768;
            const int p0 = t + 2 + PF_DIST < NT ? t + 2 + PF_DIST : NT - 1, p1 = t + 3 + PF_DIST < NT ? t + 3 + PF_DIST : NT - 1;
            LDB(B0, 0, 0); SCHED; LDA(At, 0, 0); STAGE(SA(1, 1), a1 + HTB); PREF(p0);
            WAIT_L(8); BAR; WAIT_L(0); MMA(0, 0, At, B0); BAR; SCHED;
            LDB(B1, 0, 1); STAGE(SB(0, 0), b2);
            BAR; WAIT_L(0); MMA(0, 1, At, B1); BAR;
            LDA(At, 0, 1); STAGE(SA(0, 0), a2);
            BAR; WAIT_L(0); MMA(1, 0, At, B0); BAR; SCHED;
            STAGE(SB(0, 1), b2 + HTB);
            WAIT_V(7); BAR; MMA(1, 1, At, B1); BAR;
            STAGE(SA(0, 1), a2 + HTB);
            SCHED;
            PREF(p1);
            if (wid == 7 && do_c) { if (lane == 0) __hip_atomic_fetch_add(mycnt + jc * 512, 1u, __ATOMIC_RELAXED, __HIP_MEMORY_SCOPE_AGENT); }
            if (jh == NG - 1) {
                if (wid < 4) __builtin_amdgcn_global_load_lds((const unsigned*)(ws + OFF_EW + (NG - 1) * (NLOC * 4) + (pn * 256 + wid * 64) * 4 + poff),
                                                              (LAS unsigned*)(lds + LDS_EW + wid * 256), 4, 0, 0);
            }
            SCHED;
            { OPAQUE_TID(tq);
              const unsigned cva = (unsigned)(uintptr_t)(lds + LDS_CV) + (tq >> 4) * 512 + (tq & 15) * 4;
              asm volatile("ds_read2_b32 %0, %1 offset0:0 offset1:16" : "=v"(cv[0]) : "v"(cva) : "memory");
              asm volatile("ds_read2_b32 %0, %1 offset0:32 offset1:48" : "=v"(cv[1]) : "v"(cva) : "memory");
              asm volatile("ds_read2_b32 %0, %1 offset0:64 offset1:80" : "=v"(cv[2]) : "v"(cva) : "memory");
              asm volatile("ds_read2_b32 %0, %1 offset0:96 offset1:112" : "=v"(cv[3]) : "v"(cva) : "memory"); }
            SCHED;
            LDB(B0, 1, 0); SCHED; LDA(At, 1, 0);
            WAIT_L(8); BAR;
            asm volatile("s_waitcnt lgkmcnt(0)" : "+v"(cv[0]), "+v"(cv[1]), "+v"(cv[2]), "+v"(cv[3]) :: "memory");
            float mm = 0.f;
            __builtin_amdgcn_s_setprio(1);
            MFMA16(0, 0, At, B0);
            {
#pragma unroll
                for (int r = 0; r < 8; ++r) mm = __builtin_fmaxf(mm, __builtin_fabsf(cv[r >> 1][r & 1]));
                { float m2 = mm; asm volatile("s_nop 1\n\tv_permlane16_swap_b32 %0, %1" : "+v"(mm), "+v"(m2)); mm = __builtin_fmaxf(mm, m2); }
                { float m2 = mm; asm volatile("s_nop 1\n\tv_permlane32_swap_b32 %0, %1" : "+v"(mm), "+v"(m2)); mm = __builtin_fmaxf(mm, m2); }
            }
            INTERLEAVE(1);
            asm volatile("" : "+v"(mm));
            __builtin_amdgcn_s_setprio(0);
            BAR; SCHED;
            LDB(B1, 1, 1); STAGE(SB(1, 0), b3);
            { OPAQUE_TID(tq);
              const unsigned ra = (unsigned)(uintptr_t)(lds + LDS_CTL + 128) + (tq & 15) * 4;
              if ((tq & 63) < 16) asm volatile("ds_max_u32 %0, %1" :: "v"(ra), "v"(mm) : "memory"); }
            BAR; WAIT_L(0); MMA(0, 1, At, B1); BAR;
            LDA(At, 1, 1); STAGE(SA(1, 0), a3);
            BAR; WAIT_L(0); MMA(1, 0, At, B0); BAR; SCHED;
            STAGE(SB(1, 1), b3 + HTB);
            if (wid == 7 && do_c) WAIT_V(8); else WAIT_V(7);
            BAR; MMA(1, 1, At, B1); BAR;
        }
    }
    STAMP(3);
    WAIT_V(0);
    if (wr == 0) BAR;
    dead = dead || (unsigned)__builtin_amdgcn_readfirstlane(__hip_atomic_load(tmo, __ATOMIC_RELAXED, __HIP_MEMORY_SCOPE_AGENT)) == call;
    int tz = threadIdx.x; asm volatile("" : "+v"(tz));
    const int efr = tz & 15, efq = (tz >> 4) & 3;
    float* outp = p.out + (size_t)(src * MLOC + mt * 256 + wr * 64 + efr) * NLOC + pn * 256 + wc * 32 + 4 * efq;
    const __amdgpu_buffer_rsrc_t ors = __builtin_amdgcn_make_buffer_rsrc(p.out, 0, NR * MLOC * NLOC * 4, 0x00020000);
    const LAS int* ex = (const LAS int*)(lds + LDS_EX);
    const LAS int* ew = (const LAS int*)(lds + LDS_EW);
    i32x4 ce[2][2];
#pragma unroll
    for (int bj = 0; bj < 2; ++bj)
#pragma unroll
        for (int n = 0; n < 2; ++n) ce[bj][n] = *(const LAS i32x4*)(ew + bj * HALF + wc * 32 + n * 16 + 4 * efq);
    if (dead) {
        const float qnan = __builtin_nanf("");
        for (int r = 0; r < 8; ++r) for (int c = 0; c < 4; ++c)
            *(f32x4*)(outp + (size_t)((r >> 2) * HALF + (r & 3) * 16) * NLOC + (c >> 1) * HALF + (c & 1) * 16) = (f32x4){qnan, qnan, qnan, qnan};
        return;
    }
#pragma unroll
    for (int ai = 0; ai < 2; ++ai)
#pragma unroll
        for (int m = 0; m < 4; ++m) {
            const int re = ex[ai * HALF + wr * 64 + m * 16 + efr];
            float* rowp = outp + (size_t)(ai * HALF + m * 16) * NLOC;
#pragma unroll
            for (int bj = 0; bj < 2; ++bj)
#pragma unroll
                for (int n = 0; n < 2; ++n) {
                    f32x4 o;
#pragma unroll
                    for (int j = 0; j < 4; ++j) o[j] = silu(__builtin_amdgcn_ldexpf(acc[ai][bj][m][n][j], re + ce[bj][n][j]));
                    __builtin_amdgcn_raw_buffer_store_b128(__builtin_bit_cast(u32x4, o), ors, (unsigned)((char*)(rowp + bj * HALF + n * 16) - (char*)p.out), 0, 17);
                }
        }
}

extern "C" void kernel_launch(int rank, int nranks, ncclComm_t comm,
                              hipStream_t stream,
                              void* const* d_in, const int* in_sizes, int n_in,
                              void* d_out, int out_size,
                              const int* in_dtypes, int out_dtype,
                              void* const* d_out_all,
                              void* const* d_ws_all, size_t ws_bytes) {
    if (nranks != NR || (unsigned)rank >= (unsigned)NR || ws_bytes < WS_NEEDED || n_in < 2
        || in_sizes[0] != MLOC * KD || in_sizes[1] != KD * NLOC || out_size != NR * MLOC * NLOC) {
        fprintf(stderr, "kernel_launch: unexpected problem shape\n");
        return;
    }
    static bool attr_set = false;
    if (!attr_set) {
        HIP_CHECK(hipFuncSetAttribute((const void*)ag_gemm_silu, hipFuncAttributeMaxDynamicSharedMemorySize, LDS_TOTAL));
        attr_set = true;
    }
    Params p;
    p.x = (const float*)d_in[0];
    p.w = (const float*)d_in[1];
    p.out = (float*)d_out;
    for (int q = 0; q < NR; ++q) p.ws[q] = (char*)d_ws_all[q];
    p.rank = rank;
    ag_gemm_silu<<<NWG, 512, LDS_TOTAL, stream>>>(p);
    HIP_CHECK(hipGetLastError());
}
```

```cpp
#include <hip/hip_runtime.h>
#include <rccl/rccl.h>
#include <cstdio>
#include <cstdint>

#define HIP_CHECK(expr) do { hipError_t _e = (expr); if (_e != hipSuccess) { \
    fprintf(stderr, "HIP error %s at %s:%d\n", hipGetErrorString(_e), __FILE__, __LINE__); \
    return; } } while (0)

#define LAS __attribute__((address_space(3)))
typedef _Float16 f16x8 __attribute__((ext_vector_type(8)));
typedef float f32x4 __attribute__((ext_vector_type(4)));
typedef unsigned u32x4 __attribute__((ext_vector_type(4)));
typedef int i32x4 __attribute__((ext_vector_type(4)));
typedef float f32x2 __attribute__((ext_vector_type(2)));

constexpr int NR = 8;
constexpr int KD = 4096;
constexpr int NLOC = 4096;
constexpr int MLOC = 512;
constexpr int BK = 64, HALF = 128;
constexpr int HTB = HALF * BK * 2;
constexpr int STAGE_BYTES = 8 * HTB;
constexpr int NT = KD / BK;
constexpr int NWG = 256;
constexpr int PF_DIST = 3;
constexpr int NG = 16;
constexpr int D = 4;

constexpr size_t OFF_CTR_CALL = 0, OFF_CTR_X = 256, OFF_TMO = 512, OFF_FLAG = 4096;
constexpr size_t OFF_CNT = 8192;
constexpr int XR = 8;
constexpr size_t OFF_EX = 512u << 10, EX_BYTES = 4096;
constexpr size_t OFF_W16 = 1u << 20;
constexpr size_t OFF_EW = 33u << 20;
constexpr size_t OFF_X16 = 34u << 20, X16_BYTES = 4u << 20;
constexpr size_t WS_NEEDED = OFF_X16 + XR * X16_BYTES;

constexpr int LDS_CV = STAGE_BYTES;
constexpr int LDS_EX = LDS_CV + 16384;
constexpr int LDS_EW = LDS_EX + 1024;
constexpr int LDS_PF = LDS_EW + 2048;
constexpr int LDS_POLL = LDS_PF + 2048;
constexpr int LDS_RED = LDS_POLL + 2048;
constexpr int LDS_CTL = LDS_RED + 512;
constexpr int LDS_TOTAL = LDS_CTL + 512;

constexpr unsigned long long DEADLINE_TICKS = 100000000ull;

struct Params {
    const float* x;
    const float* w;
    float* out;
    char* ws[NR];
    int rank;
};

__device__ __forceinline__ int lds_byte(int r, int c) {
    const int st = (r >> 4) * 2 + (c >> 5), rr = r & 15, cc = c & 31, ob = rr * 64 + cc * 2;
    return st * 1024 + (ob ^ (((ob >> 9) & 1) << 5));
}
__device__ __forceinline__ unsigned w16_off(int n, int k) {
    return (unsigned)OFF_W16 + (unsigned)(((n >> 8) * NT + (k >> 6)) * 2 + ((n >> 7) & 1)) * HTB + lds_byte(n & 127, k & 63);
}

__device__ __forceinline__ int scale_exp(float amax) {
    return amax > 0.f ? __builtin_amdgcn_frexp_expf(amax) - 15 : -200;
}

__device__ __forceinline__ int grow_exp(int erun, float amax) {
    const int e = scale_exp(amax);
    return e > erun ? e + 1 : erun;
}

__device__ __forceinline__ unsigned pack2(float a, float b, int e) {
    const _Float16 ha = (_Float16)__builtin_amdgcn_ldexpf(a, -e), hb = (_Float16)__builtin_amdgcn_ldexpf(b, -e);
    return (unsigned)__builtin_bit_cast(unsigned short, ha) | ((unsigned)__builtin_bit_cast(unsigned short, hb) << 16);
}

__device__ __forceinline__ float silu(float y) {
    const float c = __builtin_fminf(__builtin_fmaxf(y, -60.f), 60.f);
    const float e = __builtin_amdgcn_exp2f(c * -1.44269504088896341f);
    return y * __builtin_amdgcn_rcpf(1.0f + e);
}

template <bool SYS>
__device__ __forceinline__ bool wait_ge(unsigned* word, unsigned want, unsigned* tmo, unsigned call, int lane) {
    const unsigned long long t0 = __builtin_amdgcn_s_memrealtime();
    for (;;) {
        const unsigned v = SYS ? __hip_atomic_load(word, __ATOMIC_RELAXED, __HIP_MEMORY_SCOPE_SYSTEM)
                               : __hip_atomic_load(word, __ATOMIC_RELAXED, __HIP_MEMORY_SCOPE_AGENT);
        if ((unsigned)__builtin_amdgcn_readfirstlane(v) >= want) return true;
        if ((unsigned)__builtin_amdgcn_readfirstlane(__hip_atomic_load(tmo, __ATOMIC_RELAXED, __HIP_MEMORY_SCOPE_AGENT)) == call
            || __builtin_amdgcn_s_memrealtime() - t0 > DEADLINE_TICKS) {
            if (lane == 0) __hip_atomic_store(tmo, call, __ATOMIC_RELAXED, __HIP_MEMORY_SCOPE_AGENT);
            return false;
        }
        __builtin_amdgcn_s_sleep(2);
    }
}

__global__ void __launch_bounds__(512, 2) ag_gemm_silu(Params p) {
    extern __shared__ __attribute__((aligned(16))) char smem[];
    LAS char* lds = (LAS char*)smem;
    const int tid = threadIdx.x, wid = __builtin_amdgcn_readfirstlane(tid >> 6), lane = tid & 63;
    const int wr = wid >> 2, wc = wid & 3, fr = lane & 15, fq = lane >> 4;
    const int b = blockIdx.x, src = b & 7, idx = b >> 3, mt = idx & 1, pn = idx >> 1;
    char* ws = p.ws[p.rank];
    LAS unsigned* ctl = (LAS unsigned*)(lds + LDS_CTL);
    LAS float* red = (LAS float*)(lds + LDS_RED);
    unsigned* tmo = (unsigned*)(ws + OFF_TMO);
    const __amdgpu_buffer_rsrc_t rs = __builtin_amdgcn_make_buffer_rsrc(ws, 0, (int)WS_NEEDED, 0x00020000);
    const int n0 = ((b & 7) * 32 + (b >> 3)) * 16;
    const char* wbase = (const char*)(p.w + n0);
    unsigned* mycnt = (unsigned*)(ws + OFF_CNT + (n0 >> 8) * 128);
    int erun;

    unsigned call;
    {
        const int row = 2 * b + (tid >> 8), kk = (tid & 255) * 16, cq = tid & 3, cg = tid >> 2;
        const f32x4* xsrc = (const f32x4*)(p.x + (size_t)row * KD + kk);
        f32x4 v[4], wv[8];
#pragma unroll
        for (int i = 0; i < 4; ++i) v[i] = xsrc[i];
        asm volatile("" ::: "memory");
        if (tid == 0) {
            const unsigned t = __hip_atomic_fetch_add((unsigned*)(ws + OFF_CTR_CALL), 1u, __ATOMIC_RELAXED, __HIP_MEMORY_SCOPE_AGENT);
            ctl[0] = t / NWG + 1;
        }
        float m = 0.f;
#pragma unroll
        for (int i = 0; i < 4; ++i)
#pragma unroll
            for (int j = 0; j < 4; ++j) m = __builtin_fmaxf(m, __builtin_fabsf(v[i][j]));
#pragma unroll
        for (int s = 1; s < 64; s <<= 1) m = __builtin_fmaxf(m, __shfl_xor(m, s));
        LAS float* redx = (LAS float*)(lds + LDS_CTL + 64);
        if (lane == 0) redx[wid] = m;
        __syncthreads();
        const int w0 = (tid >> 8) * 4;
        m = __builtin_fmaxf(__builtin_fmaxf(redx[w0], redx[w0 + 1]), __builtin_fmaxf(redx[w0 + 2], redx[w0 + 3]));
        call = __builtin_amdgcn_readfirstlane(ctl[0]);
        const int e = scale_exp(m);
        const unsigned ioff = (unsigned)(OFF_X16 + (call % XR) * X16_BYTES) + (unsigned)(kk >> 6) * 65536u + (unsigned)(row >> 7) * HTB;
        const int kl = kk & 63;
#pragma unroll
        for (int c = 0; c < 2; ++c) {
            u32x4 o;
            o.x = pack2(v[2 * c][0], v[2 * c][1], e); o.y = pack2(v[2 * c][2], v[2 * c][3], e);
            o.z = pack2(v[2 * c + 1][0], v[2 * c + 1][1], e); o.w = pack2(v[2 * c + 1][2], v[2 * c + 1][3], e);
            __builtin_amdgcn_raw_buffer_store_b128(o, rs, ioff + lds_byte(row & 127, kl + 8 * c), 0, 17);
        }
        if ((tid & 255) == 0)
            __builtin_amdgcn_raw_buffer_store_b32((unsigned)e, rs, (unsigned)(OFF_EX + (call % XR) * EX_BYTES) + row * 4, 0, 17);
        asm volatile("" ::: "memory");
        if (wid != 0) {
#pragma unroll
            for (int j = 0; j < 8; ++j) wv[j] = *(const f32x4*)(wbase + (size_t)(cg * 8 + j) * (NLOC * 4) + cq * 16);
            asm volatile("s_waitcnt vmcnt(8)" ::: "memory");
        } else {
            asm volatile("s_waitcnt vmcnt(0)" ::: "memory");
        }
        __builtin_amdgcn_s_barrier();
        asm volatile("" ::: "memory");
        if (wid == 0) {
            unsigned xticket = 0;
            if (tid == 0) xticket = __hip_atomic_fetch_add((unsigned*)(ws + OFF_CTR_X), 1u, __ATOMIC_RELAXED, __HIP_MEMORY_SCOPE_AGENT);
            asm volatile("" ::: "memory");
#pragma unroll
            for (int j = 0; j < 8; ++j) wv[j] = *(const f32x4*)(wbase + (size_t)(cg * 8 + j) * (NLOC * 4) + cq * 16);
            if (tid == 0 && xticket % NWG == NWG - 1) {
#pragma unroll
                for (int r = 0; r < NR; ++r)
                    __hip_atomic_store((unsigned*)(p.ws[r] + OFF_FLAG + 128 * p.rank), call, __ATOMIC_RELAXED, __HIP_MEMORY_SCOPE_SYSTEM);
            }
        }
        float cm[4] = {0.f, 0.f, 0.f, 0.f};
#pragma unroll
        for (int j = 0; j < 8; ++j)
#pragma unroll
            for (int c = 0; c < 4; ++c) cm[c] = __builtin_fmaxf(cm[c], __builtin_fabsf(wv[j][c]));
#pragma unroll
        for (int c = 0; c < 4; ++c)
#pragma unroll
            for (int s = 4; s < 64; s <<= 1) cm[c] = __builtin_fmaxf(cm[c], __shfl_xor(cm[c], s));
        if (lane < 4) {
#pragma unroll
            for (int c = 0; c < 4; ++c) red[(4 * lane + c) * 8 + wid] = cm[c];
        }
        __syncthreads();
        int e4[4];
#pragma unroll
        for (int c = 0; c < 4; ++c) {
            float mm = 0.f;
#pragma unroll
            for (int w = 0; w < 8; ++w) mm = __builtin_fmaxf(mm, red[(4 * cq + c) * 8 + w]);
            e4[c] = grow_exp(-300, mm);
        }
        {
            float mm = 0.f;
#pragma unroll
            for (int w = 0; w < 8; ++w) mm = __builtin_fmaxf(mm, red[(tid & 15) * 8 + w]);
            erun = grow_exp(-300, mm);
            if (tid < 16) *(LAS float*)(lds + LDS_CTL + 128 + tid * 4) = mm;
        }
        const unsigned woff = w16_off(n0 + 4 * cq, cg * 8);
#pragma unroll
        for (int c = 0; c < 4; ++c) {
            u32x4 o;
            o.x = pack2(wv[0][c], wv[1][c], e4[c]); o.y = pack2(wv[2][c], wv[3][c], e4[c]);
            o.z = pack2(wv[4][c], wv[5][c], e4[c]); o.w = pack2(wv[6][c], wv[7][c], e4[c]);
            __builtin_amdgcn_raw_buffer_store_b128(o, rs, woff + c * 64, 0, 16);
        }
        if (tid < 16) __builtin_amdgcn_raw_buffer_store_b32((unsigned)erun, rs, (unsigned)OFF_EW + (D - 1) * (NLOC * 4) + (n0 + tid) * 4, 0, 16);
        asm volatile("s_waitcnt vmcnt(0)" ::: "memory");
        __syncthreads();
        if (tid == 64) __hip_atomic_fetch_add(mycnt + (D - 1) * 512, 1u, __ATOMIC_RELAXED, __HIP_MEMORY_SCOPE_AGENT);
    }
    int jnext = D;
    {
        unsigned* flag = (unsigned*)(ws + OFF_FLAG + 128 * src);
        for (; jnext < NG; ++jnext) {
            if (wid == 0) {
                unsigned ok = 0;
                for (int i = 0; i < 4 && !ok; ++i) {
                    ok = (unsigned)__builtin_amdgcn_readfirstlane(__hip_atomic_load(flag, __ATOMIC_RELAXED, __HIP_MEMORY_SCOPE_SYSTEM)) >= call;
                    if (!ok) __builtin_amdgcn_s_sleep(8);
                }
                if (lane == 0) ctl[2] = ok;
            }
            __syncthreads();
            if (__builtin_amdgcn_readfirstlane(ctl[2])) break;
            float cw[8];
#pragma unroll
            for (int r = 0; r < 8; ++r) cw[r] = *(const float*)(wbase + (size_t)(jnext * 256 + (tid >> 4) * 8 + r) * (NLOC * 4) + (tid & 15) * 4);
            float mm = 0.f;
#pragma unroll
            for (int r = 0; r < 8; ++r) mm = __builtin_fmaxf(mm, __builtin_fabsf(cw[r]));
            mm = __builtin_fmaxf(mm, __shfl_xor(mm, 16));
            mm = __builtin_fmaxf(mm, __shfl_xor(mm, 32));
            LAS unsigned* rmax = (LAS unsigned*)(lds + LDS_CTL + 128) + (tid & 15);
            if (lane < 16) __hip_atomic_fetch_max(rmax, __builtin_bit_cast(unsigned, mm), __ATOMIC_RELAXED, __HIP_MEMORY_SCOPE_WORKGROUP);
            __syncthreads();
            const int enew = grow_exp(erun, __builtin_bit_cast(float, *(volatile LAS unsigned*)rmax));
            u32x4 o;
            o.x = pack2(cw[0], cw[1], enew); o.y = pack2(cw[2], cw[3], enew); o.z = pack2(cw[4], cw[5], enew); o.w = pack2(cw[6], cw[7], enew);
            __builtin_amdgcn_raw_buffer_store_b128(o, rs, w16_off(n0 + (tid & 15), jnext * 256 + (tid >> 4) * 8), 0, 16);
            if (wid == 0) {
                if (tid < 16) __builtin_amdgcn_raw_buffer_store_b32((unsigned)enew, rs, tid * 4, (unsigned)OFF_EW + jnext * (NLOC * 4) + n0 * 4, 16);
                if (__any(enew != erun)) { if (tid == 0) __hip_atomic_store(mycnt + jnext * 512 + 1, call, __ATOMIC_RELAXED, __HIP_MEMORY_SCOPE_AGENT); }
            }
            erun = enew;
            asm volatile("s_waitcnt vmcnt(0)" ::: "memory");
            __syncthreads();
            if (tid == 64) __hip_atomic_fetch_add(mycnt + jnext * 512, 1u, __ATOMIC_RELAXED, __HIP_MEMORY_SCOPE_AGENT);
        }
    }
    unsigned* cntp = (unsigned*)(ws + OFF_CNT + pn * 128);
    const unsigned cnt_want = 16u * call;
    bool dead = !wait_ge<false>(cntp + (D - 1) * 512, cnt_want, tmo, call, lane);
    const char* cB = ws + OFF_W16 + (size_t)pn * (NT * 2 * HTB);
    const unsigned toff = (unsigned)tid * 16u, poff = (unsigned)lane * 4u;
    const unsigned ldsw = (unsigned)wid * 1024u;
#define SA(bf, h) (((bf) * 2 + (h)) * HTB)
#define SB(bf, h) ((4 + (bf) * 2 + (h)) * HTB)
#define STAGE(bufoff, gptr) do { _Pragma("unroll") for (int _i = 0; _i < 2; ++_i) \
        __builtin_amdgcn_global_load_lds((const unsigned*)((gptr) + _i * 8192 + toff), (LAS unsigned*)(lds + (bufoff) + ldsw + _i * 8192), 16, 0, 0); } while (0)
    STAGE(SB(0, 0), cB); STAGE(SB(0, 1), cB + HTB); STAGE(SB(1, 0), cB + 2 * HTB); STAGE(SB(1, 1), cB + 3 * HTB);
    dead = !wait_ge<true>((unsigned*)(ws + OFF_FLAG + 128 * src), call, tmo, call, lane) || dead;
    const char* xs = p.ws[dead ? p.rank : src] + OFF_X16 + (call % XR) * X16_BYTES;
    const char* cA = xs + mt * 2 * HTB;
    const char* cP = xs + (idx * 8 + wid) * 256;
    const int aoff = lds_byte(wr * 64 + fr, fq * 8), boff = lds_byte(wc * 32 + fr, fq * 8);

#define PREF(t) __builtin_amdgcn_global_load_lds((const unsigned*)(cP + (size_t)(t) * 65536 + poff), (LAS unsigned*)(lds + LDS_PF + wid * 256), 4, 0, 0)
#define LDA(dst, bf, h) do { _Pragma("unroll") for (int m = 0; m < 4; ++m) _Pragma("unroll") for (int k = 0; k < 2; ++k) \
        dst[m][k] = *(const LAS f16x8*)(lds + SA(bf, h) + aoff + m * 2048 + k * 1024); } while (0)
#define LDB(dst, bf, h) do { _Pragma("unroll") for (int n = 0; n < 2; ++n) _Pragma("unroll") for (int k = 0; k < 2; ++k) \
        dst[n][k] = *(const LAS f16x8*)(lds + SB(bf, h) + boff + n * 2048 + k * 1024); } while (0)
#define MMA(ai, bj, At, Bt) do { __builtin_amdgcn_s_setprio(1); _Pragma("unroll") for (int m = 0; m < 4; ++m) _Pragma("unroll") for (int n = 0; n < 2; ++n) \
        _Pragma("unroll") for (int k = 0; k < 2; ++k) \
        acc[ai][bj][m][n] = __builtin_amdgcn_mfma_f32_16x16x32_f16(Bt[n][k], At[m][k], acc[ai][bj][m][n], 0, 0, 0); __builtin_amdgcn_s_setprio(0); } while (0)
#define WAIT_V(n) asm volatile("s_waitcnt vmcnt(" #n ")" ::: "memory")
#define WAIT_L(n) asm volatile("s_waitcnt lgkmcnt(" #n ")" ::: "memory")
#define BAR __builtin_amdgcn_s_barrier()
#define SCHED __builtin_amdgcn_sched_barrier(0)

    f32x4 acc[2][2][4][2];
#pragma unroll
    for (int a = 0; a < 2; ++a)
#pragma unroll
        for (int c = 0; c < 2; ++c)
#pragma unroll
            for (int m = 0; m < 4; ++m)
#pragma unroll
                for (int n = 0; n < 2; ++n) acc[a][c][m][n] = (f32x4){0.f, 0.f, 0.f, 0.f};
    f16x8 At[4][2], B0[2][2], B1[2][2];
#define OPAQUE_TID(v) int v = threadIdx.x; asm volatile("" : "+v"(v))
    f32x2 cv[4];
#pragma unroll
    for (int r = 0; r < 4; ++r) cv[r] = (f32x2){0.f, 0.f};

    {
        if (wid < 4) __builtin_amdgcn_global_load_lds((const unsigned*)(xs - (call % XR) * X16_BYTES - OFF_X16 + OFF_EX + (call % XR) * EX_BYTES + (mt * 256 + wid * 64) * 4 + poff),
                                                      (LAS unsigned*)(lds + LDS_EX + wid * 256), 4, 0, 0);
    }
#pragma unroll
    for (int t = 2; t < PF_DIST; ++t) PREF(t);
    STAGE(SA(0, 0), cA); STAGE(SA(0, 1), cA + HTB); STAGE(SA(1, 0), cA + 4 * HTB);
    if (wr == 1) BAR;
    WAIT_V(2); BAR; BAR;
#define MFMA16(ai, bj, At, Bt) do { _Pragma("unroll") for (int m = 0; m < 4; ++m) _Pragma("unroll") for (int n = 0; n < 2; ++n) \
        _Pragma("unroll") for (int k = 0; k < 2; ++k) \
        acc[ai][bj][m][n] = __builtin_amdgcn_mfma_f32_16x16x32_f16(Bt[n][k], At[m][k], acc[ai][bj][m][n], 0, 0, 0); } while (0)
#define INTERLEAVE(nv) do { _Pragma("unroll") for (int q = 0; q < 16; ++q) { __builtin_amdgcn_sched_group_barrier(0x8, 1, 0); __builtin_amdgcn_sched_group_barrier(0x2, nv, 0); } } while (0)
    bool grew = false;
    for (int jh = 0; jh < NG; ++jh) {
        const int t = 4 * jh;
        const bool need1 = jh + 1 >= D && jh + 1 < NG;
        const bool do_c = jh + D - 1 >= jnext && jh + D - 1 < NG;
        const int ja = jh + D < NG ? jh + D : NG - 1, jc = jh + D - 1;
        if (grew) {
            const int* eo = (const int*)(ws + OFF_EW + (jh - 1) * (NLOC * 4)) + pn * 256 + wc * 32 + 4 * fq;
#pragma unroll
            for (int bj = 0; bj < 2; ++bj)
#pragma unroll
                for (int n = 0; n < 2; ++n) {
                    i32x4 d;
#pragma unroll
                    for (int j = 0; j < 4; ++j)
                        d[j] = __hip_atomic_load(eo + bj * HALF + n * 16 + j, __ATOMIC_RELAXED, __HIP_MEMORY_SCOPE_AGENT)
                             - __hip_atomic_load(eo + NLOC + bj * HALF + n * 16 + j, __ATOMIC_RELAXED, __HIP_MEMORY_SCOPE_AGENT);
#pragma unroll
                    for (int ai = 0; ai < 2; ++ai)
#pragma unroll
                        for (int m = 0; m < 4; ++m)
#pragma unroll
                            for (int j = 0; j < 4; ++j) acc[ai][bj][m][n][j] = __builtin_amdgcn_ldexpf(acc[ai][bj][m][n][j], d[j]);
                }
            grew = false;
        }
        if (need1) {
            const unsigned pc = *(volatile LAS unsigned*)(lds + LDS_POLL + wid * 256), pg = *(volatile LAS unsigned*)(lds + LDS_POLL + wid * 256 + 4);
            unsigned chg = __builtin_amdgcn_readfirstlane(pg);
            if ((unsigned)__builtin_amdgcn_readfirstlane(pc) < cnt_want) {
                dead = !wait_ge<false>(cntp + (jh + 1) * 512, cnt_want, tmo, call, lane) || dead;
                chg = __builtin_amdgcn_readfirstlane(__hip_atomic_load(cntp + (jh + 1) * 512 + 1, __ATOMIC_RELAXED, __HIP_MEMORY_SCOPE_AGENT));
            }
            grew = chg == call;
        }
        {
            const char* a1 = cA + (size_t)(t + 1) * 65536;
            const char* a2 = cA + (size_t)(t + 2) * 65536; const char* b2 = cB + (size_t)(t + 2) * 32768;
            const char* a3 = cA + (size_t)(t + 3) * 65536; const char* b3 = cB + (size_t)(t + 3) * 32768;
            const int p0 = t + PF_DIST < NT ? t + PF_DIST : NT - 1, p1 = t + 1 + PF_DIST < NT ? t + 1 + PF_DIST : NT - 1;
            LDB(B0, 0, 0); SCHED; LDA(At, 0, 0); STAGE(SA(1, 1), a1 + HTB); PREF(p0);
            WAIT_L(8); BAR; WAIT_L(0); MMA(0, 0, At, B0); BAR; SCHED;
            LDB(B1, 0, 1); STAGE(SB(0, 0), b2);
            BAR; WAIT_L(0); MMA(0, 1, At, B1); BAR;
            LDA(At, 0, 1); STAGE(SA(0, 0), a2);
            float mm; { OPAQUE_TID(tq); mm = *(const LAS float*)(lds + LDS_CTL + 128 + (tq & 15) * 4); }
            BAR; WAIT_L(0); MMA(1, 0, At, B0); BAR; SCHED;
            STAGE(SB(0, 1), b2 + HTB);
            WAIT_V(7); BAR;
            u32x4 o; int enew; unsigned soff;
            __builtin_amdgcn_s_setprio(1);
            MFMA16(1, 1, At, B1);
            {
                enew = do_c ? grow_exp(erun, mm) : erun;
                o.x = pack2(cv[0][0], cv[0][1], enew); o.y = pack2(cv[1][0], cv[1][1], enew); o.z = pack2(cv[2][0], cv[2][1], enew); o.w = pack2(cv[3][0], cv[3][1], enew);
                OPAQUE_TID(tc);
                soff = w16_off(n0 + (tc & 15), (tc >> 4) * 8) | (do_c ? 0u : 0xFF000000u);
            }
            INTERLEAVE(3);
            asm volatile("" : "+v"(o), "+v"(enew), "+v"(soff));
            __builtin_amdgcn_s_setprio(0);
            BAR;
            STAGE(SA(0, 1), a2 + HTB);
            SCHED;
            PREF(p1);
            { const int jp = jh + 2 < NG ? jh + 2 : NG - 1;
              __builtin_amdgcn_global_load_lds((const unsigned*)((const char*)(cntp + jp * 512) + (poff & 4)), (LAS unsigned*)(lds + LDS_POLL + wid * 256), 4, 0, 16); }
            OPAQUE_TID(tq);
            { const char* ab = wbase + (size_t)(ja * 256) * (NLOC * 4);
              const unsigned aldoff = (unsigned)(tq >> 2) * (NLOC * 4) + (tq & 3) * 16;
#pragma unroll
              for (int i = 0; i < 2; ++i)
                  __builtin_amdgcn_global_load_lds((const unsigned*)(ab + (size_t)i * (128 * NLOC * 4) + aldoff), (LAS unsigned*)(lds + LDS_CV + i * 8192 + ldsw), 16, 0, 0); }
            __builtin_amdgcn_raw_buffer_store_b128(o, rs, soff, do_c ? jc * (4 * 2 * HTB) : 0, 16);
            if (wid == 0 && do_c) {
                if (tq < 16) __builtin_amdgcn_raw_buffer_store_b32((unsigned)enew, rs, tq * 4, (unsigned)OFF_EW + jc * (NLOC * 4) + n0 * 4, 16);
                if (__any(enew != erun)) { if (tq == 0) __hip_atomic_store(mycnt + jc * 512 + 1, call, __ATOMIC_RELAXED, __HIP_MEMORY_SCOPE_AGENT); }
            }
            erun = enew;
            SCHED;
            LDB(B0, 1, 0); SCHED; LDA(At, 1, 0);
            WAIT_L(8); BAR; WAIT_L(0); MMA(0, 0, At, B0); BAR; SCHED;
            LDB(B1, 1, 1); STAGE(SB(1, 0), b3);
            BAR; WAIT_L(0); MMA(0, 1, At, B1); BAR;
            LDA(At, 1, 1); STAGE(SA(1, 0), a3);
            BAR; WAIT_L(0); MMA(1, 0, At, B0); BAR; SCHED;
            STAGE(SB(1, 1), b3 + HTB);
            if (wid == 0 && do_c) WAIT_V(12); else WAIT_V(11);
            BAR; MMA(1, 1, At, B1); BAR;
        }
        {
            const int u2 = t + 4 < NT ? t + 4 : NT - 2, u3 = u2 + 1;
            const char* a1 = cA + (size_t)(t + 3) * 65536;
            const char* a2 = cA + (size_t)u2 * 65536; const char* b2 = cB + (size_t)u2 * 32768;
            const char* a3 = cA + (size_t)u3 * 65536; const char* b3 = cB + (size_t)u3 * 32768;
            const int p0 = t + 2 + PF_DIST < NT ? t + 2 + PF_DIST : NT - 1, p1 = t + 3 + PF_DIST < NT ? t + 3 + PF_DIST : NT - 1;
            LDB(B0, 0, 0); SCHED; LDA(At, 0, 0); STAGE(SA(1, 1), a1 + HTB); PREF(p0);
            WAIT_L(8); BAR; WAIT_L(0); MMA(0, 0, At, B0); BAR; SCHED;
            LDB(B1, 0, 1); STAGE(SB(0, 0), b2);
            BAR; WAIT_L(0); MMA(0, 1, At, B1); BAR;
            LDA(At, 0, 1); STAGE(SA(0, 0), a2);
            BAR; WAIT_L(0); MMA(1, 0, At, B0); BAR; SCHED;
            STAGE(SB(0, 1), b2 + HTB);
            WAIT_V(7); BAR; MMA(1, 1, At, B1); BAR;
            STAGE(SA(0, 1), a2 + HTB);
            SCHED;
            PREF(p1);
            if (wid == 7 && do_c) { if (lane == 0) __hip_atomic_fetch_add(mycnt + jc * 512, 1u, __ATOMIC_RELAXED, __HIP_MEMORY_SCOPE_AGENT); }
            if (jh == NG - 1) {
                if (wid < 4) __builtin_amdgcn_global_load_lds((const unsigned*)(ws + OFF_EW + (NG - 1) * (NLOC * 4) + (pn * 256 + wid * 64) * 4 + poff),
                                                              (LAS unsigned*)(lds + LDS_EW + wid * 256), 4, 0, 0);
            }
            SCHED;
            { OPAQUE_TID(tq);
              const unsigned cva = (unsigned)(uintptr_t)(lds + LDS_CV) + (tq >> 4) * 512 + (tq & 15) * 4;
              asm volatile("ds_read2_b32 %0, %1 offset0:0 offset1:16" : "=v"(cv[0]) : "v"(cva) : "memory");
              asm volatile("ds_read2_b32 %0, %1 offset0:32 offset1:48" : "=v"(cv[1]) : "v"(cva) : "memory");
              asm volatile("ds_read2_b32 %0, %1 offset0:64 offset1:80" : "=v"(cv[2]) : "v"(cva) : "memory");
              asm volatile("ds_read2_b32 %0, %1 offset0:96 offset1:112" : "=v"(cv[3]) : "v"(cva) : "memory"); }
            SCHED;
            LDB(B0, 1, 0); SCHED; LDA(At, 1, 0);
            WAIT_L(8); BAR;
            asm volatile("s_waitcnt lgkmcnt(0)" : "+v"(cv[0]), "+v"(cv[1]), "+v"(cv[2]), "+v"(cv[3]) :: "memory");
            float mm = 0.f;
            __builtin_amdgcn_s_setprio(1);
            MFMA16(0, 0, At, B0);
            {
#pragma unroll
                for (int r = 0; r < 8; ++r) mm = __builtin_fmaxf(mm, __builtin_fabsf(cv[r >> 1][r & 1]));
                { float m2 = mm; asm volatile("s_nop 1\n\tv_permlane16_swap_b32 %0, %1" : "+v"(mm), "+v"(m2)); mm = __builtin_fmaxf(mm, m2); }
                { float m2 = mm; asm volatile("s_nop 1\n\tv_permlane32_swap_b32 %0, %1" : "+v"(mm), "+v"(m2)); mm = __builtin_fmaxf(mm, m2); }
            }
            INTERLEAVE(1);
            asm volatile("" : "+v"(mm));
            __builtin_amdgcn_s_setprio(0);
            BAR; SCHED;
            LDB(B1, 1, 1); STAGE(SB(1, 0), b3);
            { OPAQUE_TID(tq);
              const unsigned ra = (unsigned)(uintptr_t)(lds + LDS_CTL + 128) + (tq & 15) * 4;
              if ((tq & 63) < 16) asm volatile("ds_max_u32 %0, %1" :: "v"(ra), "v"(mm) : "memory"); }
            BAR; WAIT_L(0); MMA(0, 1, At, B1); BAR;
            LDA(At, 1, 1); STAGE(SA(1, 0), a3);
            BAR; WAIT_L(0); MMA(1, 0, At, B0); BAR; SCHED;
            STAGE(SB(1, 1), b3 + HTB);
            if (wid == 7 && do_c) WAIT_V(8); else WAIT_V(7);
            BAR; MMA(1, 1, At, B1); BAR;
        }
    }
    WAIT_V(0);
    if (wr == 0) BAR;
    dead = dead || (unsigned)__builtin_amdgcn_readfirstlane(__hip_atomic_load(tmo, __ATOMIC_RELAXED, __HIP_MEMORY_SCOPE_AGENT)) == call;
    int tz = threadIdx.x; asm volatile("" : "+v"(tz));
    const int efr = tz & 15, efq = (tz >> 4) & 3;
    float* outp = p.out + (size_t)(src * MLOC + mt * 256 + wr * 64 + efr) * NLOC + pn * 256 + wc * 32 + 4 * efq;
    const __amdgpu_buffer_rsrc_t ors = __builtin_amdgcn_make_buffer_rsrc(p.out, 0, NR * MLOC * NLOC * 4, 0x00020000);
    const LAS int* ex = (const LAS int*)(lds + LDS_EX);
    const LAS int* ew = (const LAS int*)(lds + LDS_EW);
    i32x4 ce[2][2];
#pragma unroll
    for (int bj = 0; bj < 2; ++bj)
#pragma unroll
        for (int n = 0; n < 2; ++n) ce[bj][n] = *(const LAS i32x4*)(ew + bj * HALF + wc * 32 + n * 16 + 4 * efq);
    if (dead) {
        const float qnan = __builtin_nanf("");
        for (int r = 0; r < 8; ++r) for (int c = 0; c < 4; ++c)
            *(f32x4*)(outp + (size_t)((r >> 2) * HALF + (r & 3) * 16) * NLOC + (c >> 1) * HALF + (c & 1) * 16) = (f32x4){qnan, qnan, qnan, qnan};
        return;
    }
#pragma unroll
    for (int ai = 0; ai < 2; ++ai)
#pragma unroll
        for (int m = 0; m < 4; ++m) {
            const int re = ex[ai * HALF + wr * 64 + m * 16 + efr];
            float* rowp = outp + (size_t)(ai * HALF + m * 16) * NLOC;
#pragma unroll
            for (int bj = 0; bj < 2; ++bj)
#pragma unroll
                for (int n = 0; n < 2; ++n) {
                    f32x4 o;
#pragma unroll
                    for (int j = 0; j < 4; ++j) o[j] = silu(__builtin_amdgcn_ldexpf(acc[ai][bj][m][n][j], re + ce[bj][n][j]));
                    __builtin_amdgcn_raw_buffer_store_b128(__builtin_bit_cast(u32x4, o), ors, (unsigned)((char*)(rowp + bj * HALF + n * 16) - (char*)p.out), 0, 17);
                }
        }
}

extern "C" void kernel_launch(int rank, int nranks, ncclComm_t comm,
                              hipStream_t stream,
                              void* const* d_in, const int* in_sizes, int n_in,
                              void* d_out, int out_size,
                              const int* in_dtypes, int out_dtype,
                              void* const* d_out_all,
                              void* const* d_ws_all, size_t ws_bytes) {
    if (nranks != NR || (unsigned)rank >= (unsigned)NR || ws_bytes < WS_NEEDED || n_in < 2
        || in_sizes[0] != MLOC * KD || in_sizes[1] != KD * NLOC || out_size != NR * MLOC * NLOC) {
        fprintf(stderr, "kernel_launch: unexpected problem shape\n");
        return;
    }
    static bool attr_set = false;
    if (!attr_set) {
        HIP_CHECK(hipFuncSetAttribute((const void*)ag_gemm_silu, hipFuncAttributeMaxDynamicSharedMemorySize, LDS_TOTAL));
        attr_set = true;
    }
    Params p;
    p.x = (const float*)d_in[0];
    p.w = (const float*)d_in[1];
    p.out = (float*)d_out;
    for (int q = 0; q < NR; ++q) p.ws[q] = (char*)d_ws_all[q];
    p.rank = rank;
    ag_gemm_silu<<<NWG, 512, LDS_TOTAL, stream>>>(p);
    HIP_CHECK(hipGetLastError());
}
```
